```python
import numpy as np

kernel_name = "gpu_a2a_gemm_m4096_k4096_n4096_bf16_q8"


ABSMAX_RTOL = 0.02

COMPUTE_DTYPE = "bfloat16"
EPILOGUE = "quant_int8"
IN_QUANT = None
OUT_QUANT = "int8"
IN_DTYPES = {"x": "bfloat16", "w_mat": "bfloat16"}
WORLD_SIZE = 8

def _to_bf16(x):
    u = np.ascontiguousarray(x, dtype=np.float32).view(np.uint32)
    return ((u + 0x7FFF + ((u >> 16) & 1)) & 0xFFFF0000).view(np.float32)

def setup_inputs():

    x = _to_bf16(np.random.randn(4096, 4096).astype(np.float32))
    w_mat = _to_bf16((np.random.randn(4096, 4096) / np.sqrt(4096)).astype(np.float32))
    return {"x": x, "w_mat": w_mat}

def _quant_dequant(y, amax):
    scale = amax / 127.0
    q = np.clip(np.round(y / scale), -127.0, 127.0)
    return (q * scale).astype(np.float32)
def _epilogue(y):
    return _quant_dequant(y, np.abs(y).max())

def reference(x, w_mat):
    y = (x @ w_mat).astype(np.float32)
    return _epilogue(y)
```

```cpp
#include <hip/hip_runtime.h>
#include <rccl/rccl.h>
#include <cstdio>
#include <cstdint>

#define HIP_CHECK(expr) do { hipError_t _e = (expr); if (_e != hipSuccess) { \
    fprintf(stderr, "HIP error %s at %s:%d\n", hipGetErrorString(_e), __FILE__, __LINE__); \
    return; } } while (0)

typedef __attribute__((ext_vector_type(8))) short bf16x8;
typedef __attribute__((ext_vector_type(4))) short s16x4;
typedef __attribute__((ext_vector_type(4))) float f32x4;
typedef __attribute__((ext_vector_type(4))) unsigned u32x4;
typedef unsigned long long u64;

#define LAS __attribute__((address_space(3)))

namespace {

constexpr int NR = 8;
constexpr int MLOC = 512;
constexpr int KSH = 512;
constexpr int NDIM = 4096;
constexpr int BM = 128, BN = 64, BK = 64;
constexpr int NS = 4;
constexpr int A_BYTES = BM * BK * 2;
constexpr int B_BYTES = BK * BN * 2;
constexpr int STAGE = A_BYTES + B_BYTES;
constexpr int NWG = 256;
constexpr int NSTEP = 64;
constexpr int PUSHW = 32;

constexpr size_t WS_A = 0;
constexpr size_t WS_FLAG = 4u << 20;
constexpr size_t WS_AMAX = (4u << 20) + (64u << 10);
constexpr size_t WS_CNT = (4u << 20) + (128u << 10);
constexpr size_t WS_TMO = WS_CNT + 256;
constexpr size_t WS_NEED = WS_CNT + 4096;

constexpr unsigned long long DEADLINE_TICKS = 40000000ull;

constexpr int C_RDY = 0;
constexpr int C_SEQ = 2;
constexpr int C_FAIL = 3;
constexpr int C_RED = 4;
constexpr int C_ALL = 12;

struct Params {
  const unsigned short* x;
  const unsigned short* w;
  signed char* out;
  char* ws[NR];
  int rank;
};

__device__ __forceinline__ int vm_count() {
  unsigned v = __builtin_amdgcn_s_getreg(7 | (31 << 11));
  return (int)((v & 0xF) | (((v >> 22) & 3) << 4));
}

__device__ __forceinline__ void bar() { __builtin_amdgcn_s_barrier(); }
#define WAITV(n) asm volatile("s_waitcnt vmcnt(" #n ")" ::: "memory")
#define WAITL0() asm volatile("s_waitcnt lgkmcnt(0)" ::: "memory")

__device__ __forceinline__ void lds_wr(unsigned addr, unsigned v) {
  asm volatile("ds_write_b32 %0, %1" :: "v"(addr), "v"(v) : "memory");
}
__device__ __forceinline__ unsigned lds_rd(unsigned addr) {
  unsigned v;
  asm volatile("ds_read_b32 %0, %1\n\ts_waitcnt lgkmcnt(0)" : "=&v"(v) : "v"(addr) : "memory");
  return v;
}
#define VM0_KNOWN() __builtin_amdgcn_s_waitcnt(0x0F70)

__device__ __forceinline__ void glds16(const void* g, char* l) {
  __builtin_amdgcn_global_load_lds((const __attribute__((address_space(1))) void*)g, (LAS void*)l, 16, 0, 0);
}

__global__ __launch_bounds__(384, 2) void a2a_gemm_q8(Params P) {
  __shared__ __attribute__((aligned(16))) char smem[NS * STAGE + 128];
  const int tid = threadIdx.x;
  const int wave = __builtin_amdgcn_readfirstlane(tid >> 6);
  const int lane = tid & 63;
  const int wg = blockIdx.x;
  const int rank = P.rank;
  const unsigned lds0 = (unsigned)(size_t)(LAS char*)smem;
  const unsigned ctrl = lds0 + NS * STAGE;
#define CW(i) (ctrl + 4u * (unsigned)(i))
  char* myws = P.ws[rank];

  if (wave == 4) {
    const bool active = wg < (NR - 1) * PUSHW;
    const int dst = (rank + 1 + wg / PUSHW) & (NR - 1);
    const int pw = wg % PUSHW;
    u32x4 d[16];
    const int r8 = lane >> 3, ch = lane & 7;
    if (active) {
      const unsigned short* src = P.x + (size_t)(dst * MLOC + pw * 16 + r8) * KSH + ch * 8;
#pragma unroll
      for (int s = 0; s < 8; ++s)
#pragma unroll
        for (int h = 0; h < 2; ++h)
          d[s * 2 + h] = *(const u32x4*)(src + (size_t)h * 8 * KSH + s * 64);
    }
    unsigned ticket = 0;
    if (lane == 0)
      ticket = __hip_atomic_fetch_add((unsigned*)(myws + WS_CNT), 1u, __ATOMIC_RELAXED, __HIP_MEMORY_SCOPE_AGENT);
    bar();
    VM0_KNOWN();
    const unsigned seq = (unsigned)__builtin_amdgcn_readfirstlane(ticket) / NWG + 1u;
    lds_wr(CW(C_SEQ), seq);
    if (active) {
      char* dbase = P.ws[dst] + WS_A + ((size_t)(rank * 8) * MLOC + pw * 16 + r8) * 128 + ch * 16;
#pragma unroll
      for (int s = 0; s < 8; ++s)
#pragma unroll
        for (int h = 0; h < 2; ++h) {
          char* a = dbase + (size_t)s * (MLOC * 128) + h * 1024;
          asm volatile("global_store_dwordx4 %0, %1, off sc0 sc1\n\ts_nop 1" :: "v"(a), "v"(d[s * 2 + h]) : "memory");
        }
    }
    WAITL0();
    bar();
    if (!active) return;
    unsigned* fl = (unsigned*)(P.ws[dst] + WS_FLAG) + rank * PUSHW + pw;
    int issued = 16, flagged = 0;
    while (flagged < 8) {
      int done = issued - vm_count();
      if (done > 16) done = 16;
      while (flagged < 8 && done >= 2 * (flagged + 1)) {
        if (lane == 0) {
          unsigned* f = fl + flagged * 256;
          asm volatile("global_store_dword %0, %1, off sc0 sc1" :: "v"(f), "v"(seq) : "memory");
        }
        ++flagged; ++issued;
      }
      bar();
    }
    return;
  }

  if (wave == 5) {
    lds_wr(CW(C_RDY), 0); lds_wr(CW(C_RDY + 1), 0); lds_wr(CW(C_FAIL), 0);
    WAITL0();
    bar();
    bar();
    const unsigned seq = (unsigned)__builtin_amdgcn_readfirstlane(lds_rd(CW(C_SEQ)));
    const bool own = (lane >> 3) == rank;
    const unsigned* flags = (const unsigned*)(myws + WS_FLAG) + lane * 4;
    const unsigned long long t0 = __builtin_amdgcn_s_memrealtime();
    unsigned f0 = 0, f1 = 0, f2 = 0, f3 = 0;
    int cur = 0, extra = 0;
    bool inflight = false;
    for (unsigned it = 2;; ++it) {
      if (cur < 8) {
        if (inflight && vm_count() == 0) {
          asm volatile("" : "+v"(f0), "+v"(f1), "+v"(f2), "+v"(f3));
          bool ok = own || (f0 >= seq && f1 >= seq && f2 >= seq && f3 >= seq);
          if (__all(ok)) ++cur;
          inflight = false;
        }
        if (!inflight && cur < 8) {
          const unsigned* fp = flags + cur * 256;
          f0 = __hip_atomic_load(fp + 0, __ATOMIC_RELAXED, __HIP_MEMORY_SCOPE_SYSTEM);
          f1 = __hip_atomic_load(fp + 1, __ATOMIC_RELAXED, __HIP_MEMORY_SCOPE_SYSTEM);
          f2 = __hip_atomic_load(fp + 2, __ATOMIC_RELAXED, __HIP_MEMORY_SCOPE_SYSTEM);
          f3 = __hip_atomic_load(fp + 3, __ATOMIC_RELAXED, __HIP_MEMORY_SCOPE_SYSTEM);
          inflight = true;
        }
        if ((it & 255u) == 0u) {
          if (__builtin_amdgcn_s_memrealtime() - t0 > DEADLINE_TICKS) {
            if (lane == 0)
              __hip_atomic_store((unsigned*)(myws + WS_TMO), seq, __ATOMIC_RELAXED, __HIP_MEMORY_SCOPE_AGENT);
            lds_wr(CW(C_FAIL), 1);
            cur = 8;
          }
        }
      }
      lds_wr(CW(C_RDY + (it & 1)), (unsigned)cur);
      WAITL0();
      bar();
      if (cur == 8 && ++extra == 2) break;
    }
    return;
  }

  const int xg = wg & 7, idx = wg >> 3;
  const int tm = idx & 3, tn = xg * 8 + (idx >> 2);
  const int wm = wave >> 1, wn = wave & 1;

  const int r8 = lane >> 3;
  const int chA = (lane & 7) ^ r8;
  const unsigned offA_own = (unsigned)(tm * BM + wave * 32 + r8) * 1024u + chA * 16;
  const unsigned offA_stg = (unsigned)(tm * BM + wave * 32 + r8) * 128u + chA * 16;
  unsigned offB[2];
#pragma unroll
  for (int j = 0; j < 2; ++j) {
    int k = (wave * 2 + j) * 8 + r8;
    int f = ((k >> 1) & 1) | (((k >> 3) & 1) << 1);
    int ch = (lane & 7) ^ (f << 1);
    offB[j] = (unsigned)k * (NDIM * 2) + tn * (BN * 2) + ch * 16;
  }
  const char* xown = (const char*)P.x + (size_t)rank * MLOC * 1024;
  const char* wsA = myws + WS_A;
  const char* wbase = (const char*)P.w;

  const int fr = lane & 15, fq = lane >> 4;
  unsigned aoff[2];
#pragma unroll
  for (int kk = 0; kk < 2; ++kk)
    aoff[kk] = (unsigned)(wm * 64 + fr) * 128u + (((kk * 4 + fq) ^ (lane & 7)) * 16);
  unsigned boff[2];
  {
    const int q = (lane & 15) >> 2, p = lane & 3, g = fq;
    const int f = (q >> 1) | ((g & 1) << 1);
#pragma unroll
    for (int nt = 0; nt < 2; ++nt)
      boff[nt] = A_BYTES + (unsigned)(8 * g + q) * 128u + (((wn * 2 + nt) ^ f) * 32) + p * 8;
  }

  f32x4 acc[4][2];
#pragma unroll
  for (int i = 0; i < 4; ++i)
#pragma unroll
    for (int j = 0; j < 2; ++j) acc[i][j] = f32x4{0.f, 0.f, 0.f, 0.f};

  unsigned it = 0;
  int li = 0, ci = 0, rdy = 0;
  while (ci < NSTEP) {
    const int na = li - ci;
    if (na >= 4) WAITV(18);
    else if (na == 3) WAITV(12);
    else if (na == 2) WAITV(6);
    else if (na == 1) WAITV(0);
    bar();
    unsigned rn;
    asm volatile("ds_read_b32 %0, %1" : "=v"(rn) : "v"(CW(C_RDY + (it & 1))) : "memory");
    const int li0 = li;
    int lim = 8 + 7 * rdy;
    if (lim > NSTEP) lim = NSTEP;
    if (lim > ci + NS) lim = ci + NS;
    while (li < lim) {
      int p, s;
      const char* abase;
      unsigned aoffs, astep;
      if (li < 8) {
        p = rank; s = li;
        abase = xown + s * 128; aoffs = offA_own; astep = 8 * 1024;
      } else {
        const int j = li - 8;
        s = j / 7;
        const int jj = j - s * 7;
        p = (rank + 1 + jj) & 7;
        if (jj == 0) asm volatile("buffer_inv sc0 sc1" ::: "memory");
        abase = wsA + (size_t)(p * 8 + s) * 65536; aoffs = offA_stg; astep = 1024;
      }
      char* sb = smem + (li & (NS - 1)) * STAGE;
#pragma unroll
      for (int j = 0; j < 4; ++j)
        glds16(abase + aoffs + j * astep, sb + (wave * 4 + j) * 1024 + lane * 16);
      const char* bb = wbase + (size_t)(p * KSH + s * 64) * (NDIM * 2);
#pragma unroll
      for (int j = 0; j < 2; ++j)
        glds16(bb + offB[j], sb + A_BYTES + (wave * 2 + j) * 1024 + lane * 16);
      ++li;
    }
    if (ci < li0) {
      const unsigned sb = lds0 + (unsigned)(ci & (NS - 1)) * STAGE;
      const unsigned a0 = sb + aoff[0], a1 = sb + aoff[1], b0 = sb + boff[0], b1 = sb + boff[1];
      bf16x8 a[4][2];
      s16x4 bl[2][2], bh[2][2];
      asm volatile(
          "ds_read_b64_tr_b16 %0, %8\n\t"
          "ds_read_b64_tr_b16 %1, %8 offset:512\n\t"
          "ds_read_b64_tr_b16 %2, %9\n\t"
          "ds_read_b64_tr_b16 %3, %9 offset:512\n\t"
          "ds_read_b128 %4, %10\n\t"
          "ds_read_b128 %5, %10 offset:2048\n\t"
          "ds_read_b128 %6, %10 offset:4096\n\t"
          "ds_read_b128 %7, %10 offset:6144"
          : "=&v"(bl[0][0]), "=&v"(bh[0][0]), "=&v"(bl[1][0]), "=&v"(bh[1][0]),
            "=&v"(a[0][0]), "=&v"(a[1][0]), "=&v"(a[2][0]), "=&v"(a[3][0])
          : "v"(b0), "v"(b1), "v"(a0) : "memory");
      asm volatile(
          "ds_read_b64_tr_b16 %0, %8 offset:4096\n\t"
          "ds_read_b64_tr_b16 %1, %8 offset:4608\n\t"
          "ds_read_b64_tr_b16 %2, %9 offset:4096\n\t"
          "ds_read_b64_tr_b16 %3, %9 offset:4608\n\t"
          "ds_read_b128 %4, %10\n\t"
          "ds_read_b128 %5, %10 offset:2048\n\t"
          "ds_read_b128 %6, %10 offset:4096\n\t"
          "ds_read_b128 %7, %10 offset:6144"
          : "=&v"(bl[0][1]), "=&v"(bh[0][1]), "=&v"(bl[1][1]), "=&v"(bh[1][1]),
            "=&v"(a[0][1]), "=&v"(a[1][1]), "=&v"(a[2][1]), "=&v"(a[3][1])
          : "v"(b0), "v"(b1), "v"(a1) : "memory");
      asm volatile("s_waitcnt lgkmcnt(8)"
                   : "+v"(bl[0][0]), "+v"(bh[0][0]), "+v"(bl[1][0]), "+v"(bh[1][0]),
                     "+v"(a[0][0]), "+v"(a[1][0]), "+v"(a[2][0]), "+v"(a[3][0]), "+v"(rn) :: "memory");
      __builtin_amdgcn_sched_barrier(0);
      __builtin_amdgcn_s_setprio(1);
#pragma unroll
      for (int nt = 0; nt < 2; ++nt) {
        const bf16x8 bb = bf16x8{bl[nt][0][0], bl[nt][0][1], bl[nt][0][2], bl[nt][0][3],
                                 bh[nt][0][0], bh[nt][0][1], bh[nt][0][2], bh[nt][0][3]};
#pragma unroll
        for (int mt = 0; mt < 4; ++mt)
          acc[mt][nt] = __builtin_amdgcn_mfma_f32_16x16x32_bf16(a[mt][0], bb, acc[mt][nt], 0, 0, 0);
      }
      __builtin_amdgcn_sched_barrier(0);
      asm volatile("s_waitcnt lgkmcnt(0)"
                   : "+v"(bl[0][1]), "+v"(bh[0][1]), "+v"(bl[1][1]), "+v"(bh[1][1]),
                     "+v"(a[0][1]), "+v"(a[1][1]), "+v"(a[2][1]), "+v"(a[3][1]) :: "memory");
      __builtin_amdgcn_sched_barrier(0);
#pragma unroll
      for (int nt = 0; nt < 2; ++nt) {
        const bf16x8 bb = bf16x8{bl[nt][1][0], bl[nt][1][1], bl[nt][1][2], bl[nt][1][3],
                                 bh[nt][1][0], bh[nt][1][1], bh[nt][1][2], bh[nt][1][3]};
#pragma unroll
        for (int mt = 0; mt < 4; ++mt)
          acc[mt][nt] = __builtin_amdgcn_mfma_f32_16x16x32_bf16(a[mt][1], bb, acc[mt][nt], 0, 0, 0);
      }
      __builtin_amdgcn_s_setprio(0);
      __builtin_amdgcn_sched_barrier(0);
      ++ci;
    }
    asm volatile("s_waitcnt lgkmcnt(0)" : "+v"(rn) :: "memory");
    rdy = __builtin_amdgcn_readfirstlane(rn);
    ++it;
  }

  const unsigned seq = (unsigned)__builtin_amdgcn_readfirstlane(lds_rd(CW(C_SEQ)));
  float m = 0.f;
#pragma unroll
  for (int i = 0; i < 4; ++i)
#pragma unroll
    for (int j = 0; j < 2; ++j)
#pragma unroll
      for (int r = 0; r < 4; ++r) m = fmaxf(m, fabsf(acc[i][j][r]));
#pragma unroll
  for (int o = 32; o >= 1; o >>= 1) m = fmaxf(m, __shfl_xor(m, o));
  lds_wr(CW(C_RED + wave), __float_as_uint(m));
  WAITL0();
  bar();
  if (wave == 0 && lane < NR) {
    unsigned mb = lds_rd(CW(C_RED));
    mb = max(mb, lds_rd(CW(C_RED + 1))); mb = max(mb, lds_rd(CW(C_RED + 2))); mb = max(mb, lds_rd(CW(C_RED + 3)));
    u64 g = ((u64)seq << 32) | mb;
    u64* dstp = (u64*)(P.ws[lane] + WS_AMAX) + rank * NWG + wg;
    __hip_atomic_store(dstp, g, __ATOMIC_RELAXED, __HIP_MEMORY_SCOPE_SYSTEM);
  }
  unsigned gm = 0;
  bool failed = lds_rd(CW(C_FAIL)) != 0;
  {
    const u64* gp = (const u64*)(myws + WS_AMAX) + tid;
    const unsigned long long t0 = __builtin_amdgcn_s_memrealtime();
    unsigned pass = 0;
    while (!failed) {
      bool ok = true;
      gm = 0;
#pragma unroll
      for (int k = 0; k < 8; ++k) {
        u64 x = __hip_atomic_load(gp + k * 256, __ATOMIC_RELAXED, __HIP_MEMORY_SCOPE_SYSTEM);
        ok &= (unsigned)(x >> 32) >= seq;
        gm = max(gm, (unsigned)x);
      }
      const bool wok = __all(ok);
      bool tmo = false;
      if (!wok) {
        tmo = (__builtin_amdgcn_s_memrealtime() - t0 > DEADLINE_TICKS) ||
              (__hip_atomic_load((unsigned*)(myws + WS_TMO), __ATOMIC_RELAXED, __HIP_MEMORY_SCOPE_AGENT) == seq);
      }
      lds_wr(CW(C_ALL + wave), (wok ? 1u : 0u) | (tmo ? 2u : 0u));
      WAITL0();
      bar();
      unsigned v0 = lds_rd(CW(C_ALL)), v1 = lds_rd(CW(C_ALL + 1)), v2 = lds_rd(CW(C_ALL + 2)), v3 = lds_rd(CW(C_ALL + 3));
      bar();
      const unsigned andv = v0 & v1 & v2 & v3, orv = v0 | v1 | v2 | v3;
      if (orv & 2u) {
        failed = true;
        if (tid == 0) __hip_atomic_store((unsigned*)(myws + WS_TMO), seq, __ATOMIC_RELAXED, __HIP_MEMORY_SCOPE_AGENT);
      }
      if (andv & 1u) break;
      ++pass;
      __builtin_amdgcn_s_sleep(1);
    }
  }
#pragma unroll
  for (int o = 32; o >= 1; o >>= 1) gm = max(gm, (unsigned)__shfl_xor((int)gm, o));
  lds_wr(CW(C_RED + 4 + wave), gm);
  WAITL0();
  bar();
  gm = max(max(lds_rd(CW(C_RED + 4)), lds_rd(CW(C_RED + 5))), max(lds_rd(CW(C_RED + 6)), lds_rd(CW(C_RED + 7))));
  const float amax = __uint_as_float(gm);
  float scale = amax / 127.0f;
  if (!(scale > 0.f)) scale = 1.0f;

  signed char* tile = (signed char*)smem;
#pragma unroll
  for (int mt = 0; mt < 4; ++mt)
#pragma unroll
    for (int nt = 0; nt < 2; ++nt)
#pragma unroll
      for (int r = 0; r < 4; ++r) {
        float qv = rintf(acc[mt][nt][r] / scale);
        qv = fminf(fmaxf(qv, -127.f), 127.f);
        int qi = failed ? -128 : (int)qv;
        tile[(wm * 64 + mt * 16 + fq * 4 + r) * 64 + wn * 32 + nt * 16 + fr] = (signed char)qi;
      }
  WAITL0();
  bar();
#pragma unroll
  for (int j = 0; j < 2; ++j) {
    const int c = tid + j * 256;
    const int row = c >> 2, cc = c & 3;
    u32x4 v = *(const u32x4*)(smem + row * 64 + cc * 16);
    *(u32x4*)(P.out + (size_t)(tm * BM + row) * NDIM + tn * BN + cc * 16) = v;
  }
}

}

extern "C" void kernel_launch(int rank, int nranks, ncclComm_t comm,
                              hipStream_t stream,
                              void* const* d_in, const int* in_sizes, int n_in,
                              void* d_out, int out_size,
                              const int* in_dtypes, int out_dtype,
                              void* const* d_out_all,
                              void* const* d_ws_all, size_t ws_bytes) {
  (void)comm; (void)in_sizes; (void)n_in; (void)out_size; (void)in_dtypes; (void)out_dtype; (void)d_out_all;
  if (nranks != NR || ws_bytes < WS_NEED || (unsigned)rank >= (unsigned)NR) {
    fprintf(stderr, "unexpected configuration\n");
    return;
  }
  Params P;
  P.x = (const unsigned short*)d_in[0];
  P.w = (const unsigned short*)d_in[1];
  P.out = (signed char*)d_out;
  for (int p = 0; p < NR; ++p) P.ws[p] = (char*)d_ws_all[p];
  P.rank = rank;
  hipLaunchKernelGGL(a2a_gemm_q8, dim3(NWG), dim3(384), 0, stream, P);
  HIP_CHECK(hipGetLastError());
}
```

```cpp
#include <hip/hip_runtime.h>
#include <rccl/rccl.h>
#include <cstdio>
#include <cstdint>

#define HIP_CHECK(expr) do { hipError_t _e = (expr); if (_e != hipSuccess) { \
    fprintf(stderr, "HIP error %s at %s:%d\n", hipGetErrorString(_e), __FILE__, __LINE__); \
    return; } } while (0)

typedef __attribute__((ext_vector_type(8))) short bf16x8;
typedef __attribute__((ext_vector_type(4))) short s16x4;
typedef __attribute__((ext_vector_type(4))) float f32x4;
typedef __attribute__((ext_vector_type(4))) unsigned u32x4;
typedef unsigned long long u64;

#define LAS __attribute__((address_space(3)))

namespace {

constexpr int NR = 8;
constexpr int MLOC = 512;
constexpr int KSH = 512;
constexpr int NDIM = 4096;
constexpr int BM = 128, BN = 128, BK = 64;
constexpr int NS = 4;
constexpr int A_BYTES = BM * BK * 2;
constexpr int B_BYTES = BK * BN * 2;
constexpr int STAGE = A_BYTES + B_BYTES;
constexpr int NWG = 256;
constexpr int NSTEP = 32;
constexpr int NOWN = 4;
constexpr int NREM = NSTEP - NOWN;
constexpr int PUSHW = 32;
constexpr int NTHREADS = 640;
#define PUSH_T 180

constexpr size_t WS_A = 0;
constexpr size_t WS_FLAG = 4u << 20;
constexpr size_t WS_AMAX = (4u << 20) + (64u << 10);
constexpr size_t WS_CNT = (4u << 20) + (128u << 10);
constexpr size_t WS_TMO = WS_CNT + 256;
constexpr size_t WS_XFLAG = (4u << 20) + (192u << 10);
constexpr size_t WS_PART = (6u << 20);
constexpr size_t WS_NEED = WS_PART + (8u << 20);
#define STAMP(i) do { } while (0)

constexpr unsigned long long DEADLINE_TICKS = 40000000ull;

constexpr int C_FULL = 0;
constexpr int C_DONE = 4;
constexpr int C_RDY = 8;
constexpr int C_SEQ = 9;
constexpr int C_FAIL = 10;
constexpr int C_RED = 12;
constexpr int C_ALL = 20;
constexpr int C_QUEUE = 32;

struct Params {
  const unsigned short* x;
  const unsigned short* w;
  signed char* out;
  char* ws[NR];
  int rank;
};

__device__ __forceinline__ void bar() { __builtin_amdgcn_s_barrier(); }
#define WAITV(n) asm volatile("s_waitcnt vmcnt(" #n ")" ::: "memory")
#define WAITL0() asm volatile("s_waitcnt lgkmcnt(0)" ::: "memory")

__device__ __forceinline__ void lds_wr(unsigned addr, unsigned v) {
  asm volatile("ds_write_b32 %0, %1" :: "v"(addr), "v"(v) : "memory");
}
__device__ __forceinline__ unsigned lds_rd(unsigned addr) {
  unsigned v;
  asm volatile("ds_read_b32 %0, %1\n\ts_waitcnt lgkmcnt(0)" : "=&v"(v) : "v"(addr) : "memory");
  return v;
}
__device__ __forceinline__ int lds_rd_min4(unsigned addr) {
  u32x4 v;
  asm volatile("ds_read_b128 %0, %1\n\ts_waitcnt lgkmcnt(0)" : "=&v"(v) : "v"(addr) : "memory");
  unsigned m = min(min(v[0], v[1]), min(v[2], v[3]));
  return __builtin_amdgcn_readfirstlane((int)m);
}
#define VM0_KNOWN() __builtin_amdgcn_s_waitcnt(0x0F70)

__device__ __forceinline__ void glds16(const void* g, char* l) {
  __builtin_amdgcn_global_load_lds((const __attribute__((address_space(1))) void*)g, (LAS void*)l, 16, 0, 0);
}

__global__ __launch_bounds__(NTHREADS, 3) void a2a_gemm_q8(Params P) {
  __shared__ __attribute__((aligned(16))) char smem[NS * STAGE + 512];
  const int tid = threadIdx.x;
  const int wave = __builtin_amdgcn_readfirstlane(tid >> 6);
  const int lane = tid & 63;
  const int wg = blockIdx.x;
  const int rank = P.rank;
  const unsigned lds0 = (unsigned)(size_t)(LAS char*)smem;
  const unsigned ctrl = lds0 + NS * STAGE;
#define CW(i) (ctrl + 4u * (unsigned)(i))
  char* myws = P.ws[rank];
  const int xg = wg & 7, idx = wg >> 3;
  const int hk = idx & 1;
  const int tm = (idx >> 1) & 3, tn = xg * 4 + (idx >> 3);

  if (wave == 8) {
    const unsigned long long tstart = __builtin_amdgcn_s_memrealtime();
    const bool active = wg < (NR - 1) * PUSHW;
    const int dst = (rank + 1 + wg / PUSHW) & (NR - 1);
    const int pw = wg % PUSHW;
    const int ps = pw >> 2, ptm = pw & 3;
    u32x4 d[16];
    const int r8 = lane >> 3, ch = lane & 7;
    if (active) {
      const unsigned short* src = P.x + (size_t)(dst * MLOC + ptm * 128 + r8) * KSH + ps * 64 + ch * 8;
#pragma unroll
      for (int j = 0; j < 16; ++j) d[j] = *(const u32x4*)(src + (size_t)j * 8 * KSH);
    }
    bar();
    if (!active) return;
    VM0_KNOWN();
    while (__builtin_amdgcn_s_memrealtime() - tstart < (unsigned long long)(ps * PUSH_T)) __builtin_amdgcn_s_sleep(2);
    char* dbase = P.ws[dst] + WS_A + ((size_t)(rank * 8 + ps) * MLOC + ptm * 128 + r8) * 128 + ch * 16;
#pragma unroll
    for (int j = 0; j < 16; ++j) {
      char* a_ = dbase + j * 1024;
      asm volatile("global_store_dwordx4 %0, %1, off sc0 sc1\n\ts_nop 1" :: "v"(a_), "v"(d[j]) : "memory");
    }
    WAITV(0);
    unsigned seq;
    do { seq = (unsigned)__builtin_amdgcn_readfirstlane(lds_rd(CW(C_SEQ))); } while (seq == 0);
    unsigned* fl = (unsigned*)(P.ws[dst] + WS_FLAG) + ptm * 64 + rank * 8 + ps;
    if (lane == 0) asm volatile("global_store_dword %0, %1, off sc0 sc1" :: "v"(fl), "v"(seq) : "memory");
    return;
  }

  if (wave == 9) {
    if (lane < 12) lds_wr(CW(lane), 0);
    WAITL0();
    bar();
    unsigned ticket = 0;
    if (lane == 0)
      ticket = __hip_atomic_fetch_add((unsigned*)(myws + WS_CNT), 1u, __ATOMIC_RELAXED, __HIP_MEMORY_SCOPE_AGENT);
    const unsigned seq = (unsigned)__builtin_amdgcn_readfirstlane(ticket) / NWG + 1u;
    lds_wr(CW(C_SEQ), seq);
    const unsigned* fp = (const unsigned*)(myws + WS_FLAG) + tm * 64 + lane;
    const unsigned long long t0 = __builtin_amdgcn_s_memrealtime();
    unsigned long long seen = (0xFFull << (rank * 8)) | (hk ? 0x5555555555555555ull : 0xAAAAAAAAAAAAAAAAull);
    int cnt = 0;
    while (cnt < NREM) {
      const unsigned mn = __hip_atomic_load(fp, __ATOMIC_RELAXED, __HIP_MEMORY_SCOPE_SYSTEM);
      const unsigned long long rdy = __ballot(mn >= seq);
      const unsigned long long nw = rdy & ~seen;
      if (nw) {
        if ((nw >> lane) & 1ull) {
          const int pos = cnt + __popcll(nw & ((1ull << lane) - 1ull));
          lds_wr(CW(C_QUEUE + pos), (unsigned)lane);
        }
        WAITL0();
        cnt += __popcll(nw);
        seen |= nw;
        lds_wr(CW(C_RDY), (unsigned)cnt);
      } else {
        if (__builtin_amdgcn_s_memrealtime() - t0 > DEADLINE_TICKS ||
            __hip_atomic_load((unsigned*)(myws + WS_TMO), __ATOMIC_RELAXED, __HIP_MEMORY_SCOPE_AGENT) == seq) {
          if (lane == 0)
            __hip_atomic_store((unsigned*)(myws + WS_TMO), seq, __ATOMIC_RELAXED, __HIP_MEMORY_SCOPE_AGENT);
          lds_wr(CW(C_FAIL), 1);
          if (lane < NREM) lds_wr(CW(C_QUEUE + lane), (unsigned)(rank * 8));
          WAITL0();
          lds_wr(CW(C_RDY), NREM);
          cnt = NREM;
        }
      }
    }
    return;
  }

  if (wave >= 4) {
    const int lw = wave - 4;
    const int r8 = lane >> 3;
    const int chA = (lane & 7) ^ r8;
    const unsigned offA_own = (unsigned)(tm * BM + lw * 32 + r8) * 1024u + chA * 16;
    const unsigned offA_stg = (unsigned)(tm * BM + lw * 32 + r8) * 128u + chA * 16;
    unsigned offB[4];
#pragma unroll
    for (int j = 0; j < 4; ++j) {
      const int k = (lw * 4 + j) * 4 + (lane >> 4);
      const int f = (k & 3) | (((k >> 3) & 1) << 2);
      const int sl = lane & 15;
      const int ch = (((sl >> 1) ^ f) << 1) | (sl & 1);
      offB[j] = (unsigned)k * (NDIM * 2) + tn * (BN * 2) + ch * 16;
    }
    const char* xown = (const char*)P.x + (size_t)rank * MLOC * 1024;
    const char* wsA = myws + WS_A;
    const char* wbase = (const char*)P.w;
    bar();
    int li = 0, pub = 0, slot = 0, avail = NOWN, mind = 0, unext = 0;
    while (li < NSTEP) {
      while (li >= avail || li >= mind + NS) {
        if (li - pub == 2) { WAITV(8); ++pub; lds_wr(CW(C_FULL + lw), (unsigned)pub); }
        else if (li - pub == 1) { WAITV(0); ++pub; lds_wr(CW(C_FULL + lw), (unsigned)pub); }
        mind = lds_rd_min4(CW(C_DONE));
        avail = NOWN + __builtin_amdgcn_readfirstlane((int)lds_rd(CW(C_RDY)));
        unext = __builtin_amdgcn_readfirstlane((int)lds_rd(CW(C_QUEUE + (li >= NOWN ? li - NOWN : 0))));
      }
      int p, s;
      const char* abase;
      unsigned aoffs, astep;
      if (li < NOWN) {
        p = rank; s = 2 * li + hk;
        abase = xown + s * 128; aoffs = offA_own; astep = 8 * 1024;
      } else {
        p = unext >> 3; s = unext & 7;
        abase = wsA + (size_t)unext * 65536; aoffs = offA_stg; astep = 1024;
      }
      char* sb = smem + slot * STAGE;
#pragma unroll
      for (int j = 0; j < 4; ++j)
        glds16(abase + aoffs + j * astep, sb + (lw * 4 + j) * 1024 + lane * 16);
      const char* bb = wbase + (size_t)(p * KSH + s * 64) * (NDIM * 2);
#pragma unroll
      for (int j = 0; j < 4; ++j)
        glds16(bb + offB[j], sb + A_BYTES + (lw * 4 + j) * 1024 + lane * 16);
      ++li;
      slot = (slot == NS - 1) ? 0 : slot + 1;
      u32x4 vd; unsigned vr, vq;
      asm volatile("ds_read_b128 %0, %3\n\tds_read_b32 %1, %4\n\tds_read_b32 %2, %5"
                   : "=&v"(vd), "=&v"(vr), "=&v"(vq)
                   : "v"(CW(C_DONE)), "v"(CW(C_RDY)), "v"(CW(C_QUEUE + (li >= NOWN ? li - NOWN : 0))) : "memory");
      if (li - pub == 3) { WAITV(16); ++pub; lds_wr(CW(C_FULL + lw), (unsigned)pub); }
      asm volatile("s_waitcnt lgkmcnt(0)" : "+v"(vd), "+v"(vr), "+v"(vq) :: "memory");
      mind = __builtin_amdgcn_readfirstlane((int)min(min(vd[0], vd[1]), min(vd[2], vd[3])));
      avail = NOWN + __builtin_amdgcn_readfirstlane((int)vr);
      unext = __builtin_amdgcn_readfirstlane((int)vq);
    }
    WAITV(8); lds_wr(CW(C_FULL + lw), (unsigned)(NSTEP - 1));
    WAITV(0); lds_wr(CW(C_FULL + lw), (unsigned)NSTEP);
    return;
  }

  const int wm = wave >> 1, wn = wave & 1;
  const int fr = lane & 15, fq = lane >> 4;
  unsigned aoff[2];
#pragma unroll
  for (int kk = 0; kk < 2; ++kk)
    aoff[kk] = lds0 + (unsigned)(wm * 64 + fr) * 128u + (((kk * 4 + fq) ^ (lane & 7)) * 16);
  unsigned boff[4];
  {
    const int q = (lane & 15) >> 2, p = lane & 3, g = fq;
    const int f = q | ((g & 1) << 2);
#pragma unroll
    for (int nt = 0; nt < 4; ++nt)
      boff[nt] = lds0 + A_BYTES + (unsigned)(8 * g + q) * 256u + (((wn * 4 + nt) ^ f) * 32) + p * 8;
  }

  f32x4 acc[4][4];
#pragma unroll
  for (int i = 0; i < 4; ++i)
#pragma unroll
    for (int j = 0; j < 4; ++j) acc[i][j] = f32x4{0.f, 0.f, 0.f, 0.f};

  bar();
  if (wave == 0) STAMP(0);
  {
    bf16x8 a[4][2];
    s16x4 bl[4][2], bh[4][2];
#define RD_K0(sb_) asm volatile( \
          "ds_read_b64_tr_b16 %0, %12\n\t" \
          "ds_read_b64_tr_b16 %1, %12 offset:1024\n\t" \
          "ds_read_b64_tr_b16 %2, %13\n\t" \
          "ds_read_b64_tr_b16 %3, %13 offset:1024\n\t" \
          "ds_read_b128 %8, %16\n\t" \
          "ds_read_b128 %9, %16 offset:2048\n\t" \
          "ds_read_b128 %10, %16 offset:4096\n\t" \
          "ds_read_b128 %11, %16 offset:6144\n\t" \
          "ds_read_b64_tr_b16 %4, %14\n\t" \
          "ds_read_b64_tr_b16 %5, %14 offset:1024\n\t" \
          "ds_read_b64_tr_b16 %6, %15\n\t" \
          "ds_read_b64_tr_b16 %7, %15 offset:1024" \
          : "=&v"(bl[0][0]), "=&v"(bh[0][0]), "=&v"(bl[1][0]), "=&v"(bh[1][0]), \
            "=&v"(bl[2][0]), "=&v"(bh[2][0]), "=&v"(bl[3][0]), "=&v"(bh[3][0]), \
            "=&v"(a[0][0]), "=&v"(a[1][0]), "=&v"(a[2][0]), "=&v"(a[3][0]) \
          : "v"((sb_) + boff[0]), "v"((sb_) + boff[1]), "v"((sb_) + boff[2]), "v"((sb_) + boff[3]), "v"((sb_) + aoff[0]) : "memory")
#define RD_K1(sb_) asm volatile( \
          "ds_read_b64_tr_b16 %0, %12 offset:8192\n\t" \
          "ds_read_b64_tr_b16 %1, %12 offset:9216\n\t" \
          "ds_read_b64_tr_b16 %2, %13 offset:8192\n\t" \
          "ds_read_b64_tr_b16 %3, %13 offset:9216\n\t" \
          "ds_read_b128 %8, %16\n\t" \
          "ds_read_b128 %9, %16 offset:2048\n\t" \
          "ds_read_b128 %10, %16 offset:4096\n\t" \
          "ds_read_b128 %11, %16 offset:6144\n\t" \
          "ds_read_b64_tr_b16 %4, %14 offset:8192\n\t" \
          "ds_read_b64_tr_b16 %5, %14 offset:9216\n\t" \
          "ds_read_b64_tr_b16 %6, %15 offset:8192\n\t" \
          "ds_read_b64_tr_b16 %7, %15 offset:9216" \
          : "=&v"(bl[0][1]), "=&v"(bh[0][1]), "=&v"(bl[1][1]), "=&v"(bh[1][1]), \
            "=&v"(bl[2][1]), "=&v"(bh[2][1]), "=&v"(bl[3][1]), "=&v"(bh[3][1]), \
            "=&v"(a[0][1]), "=&v"(a[1][1]), "=&v"(a[2][1]), "=&v"(a[3][1]) \
          : "v"((sb_) + boff[0]), "v"((sb_) + boff[1]), "v"((sb_) + boff[2]), "v"((sb_) + boff[3]), "v"((sb_) + aoff[1]) : "memory")
#define WAIT_K(kk, n) asm volatile("s_waitcnt lgkmcnt(" #n ")" \
                   : "+v"(bl[0][kk]), "+v"(bh[0][kk]), "+v"(bl[1][kk]), "+v"(bh[1][kk]), \
                     "+v"(bl[2][kk]), "+v"(bh[2][kk]), "+v"(bl[3][kk]), "+v"(bh[3][kk]), \
                     "+v"(a[0][kk]), "+v"(a[1][kk]), "+v"(a[2][kk]), "+v"(a[3][kk]) :: "memory")
#define BFRAG(nt, kk) bf16x8{bl[nt][kk][0], bl[nt][kk][1], bl[nt][kk][2], bl[nt][kk][3], \
                             bh[nt][kk][0], bh[nt][kk][1], bh[nt][kk][2], bh[nt][kk][3]}
#define MMA(nt, kk) do { const bf16x8 bb_ = BFRAG(nt, kk); _Pragma("unroll") for (int mt = 0; mt < 4; ++mt) \
        acc[mt][nt] = __builtin_amdgcn_mfma_f32_16x16x32_bf16(bb_, a[mt][kk], acc[mt][nt], 0, 0, 0); } while (0)
    int minf = 0;
    unsigned sb = 0;
    while (minf <= 0) minf = lds_rd_min4(CW(C_FULL));
    RD_K0(sb);
    RD_K1(sb);
    for (int ci = 0; ci < NSTEP; ++ci) {
      WAIT_K(0, 12);
      __builtin_amdgcn_sched_barrier(0);
      MMA(0, 0); MMA(1, 0); MMA(2, 0); MMA(3, 0);
      __builtin_amdgcn_sched_barrier(0);
      WAIT_K(1, 0);
      lds_wr(CW(C_DONE + wave), (unsigned)(ci + 1));
      sb = (sb == (NS - 1) * STAGE) ? 0u : sb + STAGE;
      const bool more = ci + 1 < NSTEP;
      if (more) {
        while (minf <= ci + 1) minf = lds_rd_min4(CW(C_FULL));
        RD_K0(sb);
      }
      __builtin_amdgcn_sched_barrier(0);
      MMA(0, 1); MMA(1, 1); MMA(2, 1); MMA(3, 1);
      __builtin_amdgcn_sched_barrier(0);
      if (more) RD_K1(sb);
    }
  }

  if (wave == 0) STAMP(2);
  const unsigned seq = (unsigned)__builtin_amdgcn_readfirstlane(lds_rd(CW(C_SEQ)));
  bool failed = lds_rd(CW(C_FAIL)) != 0;
  if (wm != hk) {
    char* pb = myws + WS_PART + ((size_t)(wg * 2 + wn) * 16) * 1024 + lane * 16;
#pragma unroll
    for (int i = 0; i < 4; ++i)
#pragma unroll
      for (int j = 0; j < 4; ++j) {
        char* a_ = pb + (i * 4 + j) * 1024;
        asm volatile("global_store_dwordx4 %0, %1, off sc1\n\ts_nop 1" :: "v"(a_), "v"(acc[i][j]) : "memory");
      }
    WAITV(0);
    if (lane == 0)
      __hip_atomic_store((unsigned*)(myws + WS_XFLAG + (size_t)(wg * 2 + wn) * 128), seq, __ATOMIC_RELAXED, __HIP_MEMORY_SCOPE_AGENT);
    return;
  }
  {
    const int pwg = wg ^ 8;
    const unsigned* xf = (const unsigned*)(myws + WS_XFLAG + (size_t)(pwg * 2 + wn) * 128);
    const unsigned long long t0 = __builtin_amdgcn_s_memrealtime();
    while (!failed && __hip_atomic_load(xf, __ATOMIC_RELAXED, __HIP_MEMORY_SCOPE_AGENT) < seq) {
      if (__builtin_amdgcn_s_memrealtime() - t0 > DEADLINE_TICKS ||
          __hip_atomic_load((unsigned*)(myws + WS_TMO), __ATOMIC_RELAXED, __HIP_MEMORY_SCOPE_AGENT) == seq) {
        __hip_atomic_store((unsigned*)(myws + WS_TMO), seq, __ATOMIC_RELAXED, __HIP_MEMORY_SCOPE_AGENT);
        failed = true;
      }
    }
    const char* pb = myws + WS_PART + ((size_t)(pwg * 2 + wn) * 16) * 1024 + lane * 16;
    f32x4 t[16];
#pragma unroll
    for (int i = 0; i < 16; ++i) {
      const char* a_ = pb + i * 1024;
      asm volatile("global_load_dwordx4 %0, %1, off sc1" : "=&v"(t[i]) : "v"(a_) : "memory");
    }
    asm volatile("s_waitcnt vmcnt(0)"
                 : "+v"(t[0]), "+v"(t[1]), "+v"(t[2]), "+v"(t[3]), "+v"(t[4]), "+v"(t[5]), "+v"(t[6]), "+v"(t[7]),
                   "+v"(t[8]), "+v"(t[9]), "+v"(t[10]), "+v"(t[11]), "+v"(t[12]), "+v"(t[13]), "+v"(t[14]), "+v"(t[15]) :: "memory");
#pragma unroll
    for (int i = 0; i < 4; ++i)
#pragma unroll
      for (int j = 0; j < 4; ++j) acc[i][j] += t[i * 4 + j];
  }
  if (wn == 0) STAMP(1);

  float m = 0.f;
#pragma unroll
  for (int i = 0; i < 4; ++i)
#pragma unroll
    for (int j = 0; j < 4; ++j)
#pragma unroll
      for (int r = 0; r < 4; ++r) m = fmaxf(m, fabsf(acc[i][j][r]));
#pragma unroll
  for (int o = 32; o >= 1; o >>= 1) m = fmaxf(m, __shfl_xor(m, o));
  lds_wr(CW(C_RED + wn), __float_as_uint(m));
  lds_wr(CW(C_RED + 2 + wn), failed ? 1u : 0u);
  WAITL0();
  bar();
  failed = (lds_rd(CW(C_RED + 2)) | lds_rd(CW(C_RED + 3))) != 0;
  if (wn == 0 && lane < NR) {
    unsigned mb = max(lds_rd(CW(C_RED)), lds_rd(CW(C_RED + 1)));
    u64 g = ((u64)seq << 32) | mb;
    u64* dstp = (u64*)(P.ws[lane] + WS_AMAX) + rank * NWG + wg;
    __hip_atomic_store(dstp, g, __ATOMIC_RELAXED, __HIP_MEMORY_SCOPE_SYSTEM);
  }
  if (wn == 0) STAMP(3);
  unsigned gm = 0;
  {
    const u64* gp = (const u64*)(myws + WS_AMAX) + wn * 64 + lane;
    const unsigned long long t0 = __builtin_amdgcn_s_memrealtime();
    while (!failed) {
      bool ok = true;
      gm = 0;
#pragma unroll
      for (int k = 0; k < 16; ++k) {
        u64 x = __hip_atomic_load(gp + k * 128, __ATOMIC_RELAXED, __HIP_MEMORY_SCOPE_SYSTEM);
        ok &= (unsigned)(x >> 32) >= seq;
        gm = max(gm, (unsigned)x);
      }
      const bool wok = __all(ok);
      bool tmo = false;
      if (!wok) {
        tmo = (__builtin_amdgcn_s_memrealtime() - t0 > DEADLINE_TICKS) ||
              (__hip_atomic_load((unsigned*)(myws + WS_TMO), __ATOMIC_RELAXED, __HIP_MEMORY_SCOPE_AGENT) == seq);
      }
      lds_wr(CW(C_ALL + wn), (wok ? 1u : 0u) | (tmo ? 2u : 0u));
      WAITL0();
      bar();
      unsigned v0 = lds_rd(CW(C_ALL)), v1 = lds_rd(CW(C_ALL + 1));
      bar();
      if ((v0 | v1) & 2u) {
        failed = true;
        if (lane == 0) __hip_atomic_store((unsigned*)(myws + WS_TMO), seq, __ATOMIC_RELAXED, __HIP_MEMORY_SCOPE_AGENT);
      }
      if (v0 & v1 & 1u) break;
    }
  }
#pragma unroll
  for (int o = 32; o >= 1; o >>= 1) gm = max(gm, (unsigned)__shfl_xor((int)gm, o));
  lds_wr(CW(C_RED + 4 + wn), gm);
  WAITL0();
  bar();
  gm = max(lds_rd(CW(C_RED + 4)), lds_rd(CW(C_RED + 5)));
  const float amax = __uint_as_float(gm);
  if (wn == 0) STAMP(4);
  float scale = amax / 127.0f;
  if (!(scale > 0.f)) scale = 1.0f;
  const float rscale = 1.0f / scale;

  signed char* tile = (signed char*)smem + wn * 4096;
#pragma unroll
  for (int mt = 0; mt < 4; ++mt)
#pragma unroll
    for (int nt = 0; nt < 4; ++nt) {
      unsigned pk = 0;
#pragma unroll
      for (int r = 0; r < 4; ++r) {
        const float y = acc[mt][nt][r];
        float q0 = y * rscale;
        q0 = fmaf(fmaf(-q0, scale, y), rscale, q0);
        float qv = rintf(q0);
        qv = fminf(fmaxf(qv, -127.f), 127.f);
        int qi = failed ? -128 : (int)qv;
        pk |= ((unsigned)qi & 0xFFu) << (8 * r);
      }
      *(unsigned*)(tile + (mt * 16 + fr) * 64 + nt * 16 + fq * 4) = pk;
    }
  WAITL0();
  __builtin_amdgcn_wave_barrier();
#pragma unroll
  for (int j = 0; j < 4; ++j) {
    const int c = lane + j * 64;
    const int row = c >> 2, cc = c & 3;
    u32x4 v = *(const u32x4*)((const char*)tile + row * 64 + cc * 16);
    *(u32x4*)(P.out + (size_t)(tm * BM + hk * 64 + row) * NDIM + tn * BN + wn * 64 + cc * 16) = v;
  }
  if (wn == 0) STAMP(5);
}

}

extern "C" void kernel_launch(int rank, int nranks, ncclComm_t comm,
                              hipStream_t stream,
                              void* const* d_in, const int* in_sizes, int n_in,
                              void* d_out, int out_size,
                              const int* in_dtypes, int out_dtype,
                              void* const* d_out_all,
                              void* const* d_ws_all, size_t ws_bytes) {
  (void)comm; (void)in_sizes; (void)n_in; (void)out_size; (void)in_dtypes; (void)out_dtype; (void)d_out_all;
  if (nranks != NR || ws_bytes < WS_NEED || (unsigned)rank >= (unsigned)NR) {
    fprintf(stderr, "unexpected configuration\n");
    return;
  }
  Params P;
  P.x = (const unsigned short*)d_in[0];
  P.w = (const unsigned short*)d_in[1];
  P.out = (signed char*)d_out;
  for (int p = 0; p < NR; ++p) P.ws[p] = (char*)d_ws_all[p];
  P.rank = rank;
  hipLaunchKernelGGL(a2a_gemm_q8, dim3(NWG), dim3(NTHREADS), 0, stream, P);
  HIP_CHECK(hipGetLastError());
}
```

```cpp
#include <hip/hip_runtime.h>
#include <rccl/rccl.h>
#include <cstdio>
#include <cstdint>

#define HIP_CHECK(expr) do { hipError_t _e = (expr); if (_e != hipSuccess) { \
    fprintf(stderr, "HIP error %s at %s:%d\n", hipGetErrorString(_e), __FILE__, __LINE__); \
    return; } } while (0)

typedef __attribute__((ext_vector_type(8))) short bf16x8;
typedef __attribute__((ext_vector_type(4))) short s16x4;
typedef __attribute__((ext_vector_type(4))) float f32x4;
typedef __attribute__((ext_vector_type(4))) unsigned u32x4;
typedef unsigned long long u64;

#define LAS __attribute__((address_space(3)))

namespace {

constexpr int NR = 8;
constexpr int MLOC = 512;
constexpr int KSH = 512;
constexpr int NDIM = 4096;
constexpr int BM = 128, BN = 128, BK = 64;
constexpr int NS = 4;
constexpr int A_BYTES = BM * BK * 2;
constexpr int B_BYTES = BK * BN * 2;
constexpr int STAGE = A_BYTES + B_BYTES;
constexpr int NWG = 256;
constexpr int NSTEP = 32;
constexpr int NOWN = 4;
constexpr int NREM = NSTEP - NOWN;
constexpr int PUSHW = 32;
constexpr int NTHREADS = 640;
#define PUSH_MODE 3
#define PUSH_T 180

constexpr size_t WS_A = 0;
constexpr size_t WS_FLAG = 4u << 20;
constexpr size_t WS_AMAX = (4u << 20) + (64u << 10);
constexpr size_t WS_CNT = (4u << 20) + (128u << 10);
constexpr size_t WS_TMO = WS_CNT + 256;
constexpr size_t WS_FIN = (4u << 20) + (160u << 10);
constexpr size_t WS_XFLAG = (4u << 20) + (192u << 10);
constexpr size_t WS_PART = (6u << 20);
constexpr size_t WS_NEED = WS_PART + (4u << 20);
#define STAMP(i) do { } while (0)

constexpr unsigned long long DEADLINE_TICKS = 40000000ull;

constexpr int C_FULL = 0;
constexpr int C_DONE = 4;
constexpr int C_RDY = 8;
constexpr int C_SEQ = 9;
constexpr int C_FAIL = 10;
constexpr int C_RED = 12;
constexpr int C_ALL = 20;
constexpr int C_QUEUE = 32;

struct Params {
  const unsigned short* x;
  const unsigned short* w;
  signed char* out;
  char* ws[NR];
  int rank;
};

__device__ __forceinline__ void bar() { __builtin_amdgcn_s_barrier(); }
#define WAITV(n) asm volatile("s_waitcnt vmcnt(" #n ")" ::: "memory")
#define WAITL0() asm volatile("s_waitcnt lgkmcnt(0)" ::: "memory")

__device__ __forceinline__ void lds_wr(unsigned addr, unsigned v) {
  asm volatile("ds_write_b32 %0, %1" :: "v"(addr), "v"(v) : "memory");
}
__device__ __forceinline__ unsigned lds_rd(unsigned addr) {
  unsigned v;
  asm volatile("ds_read_b32 %0, %1\n\ts_waitcnt lgkmcnt(0)" : "=&v"(v) : "v"(addr) : "memory");
  return v;
}
__device__ __forceinline__ int lds_rd_min4(unsigned addr) {
  u32x4 v;
  asm volatile("ds_read_b128 %0, %1\n\ts_waitcnt lgkmcnt(0)" : "=&v"(v) : "v"(addr) : "memory");
  unsigned m = min(min(v[0], v[1]), min(v[2], v[3]));
  return __builtin_amdgcn_readfirstlane((int)m);
}
__device__ __forceinline__ unsigned wave_max_u32(unsigned v) {
  v = max(v, (unsigned)__builtin_amdgcn_update_dpp(0, (int)v, 0x111, 0xF, 0xF, false));
  v = max(v, (unsigned)__builtin_amdgcn_update_dpp(0, (int)v, 0x112, 0xF, 0xF, false));
  v = max(v, (unsigned)__builtin_amdgcn_update_dpp(0, (int)v, 0x114, 0xF, 0xF, false));
  v = max(v, (unsigned)__builtin_amdgcn_update_dpp(0, (int)v, 0x118, 0xF, 0xF, false));
  v = max(v, (unsigned)__builtin_amdgcn_update_dpp(0, (int)v, 0x142, 0xA, 0xF, false));
  v = max(v, (unsigned)__builtin_amdgcn_update_dpp(0, (int)v, 0x143, 0xC, 0xF, false));
  return (unsigned)__builtin_amdgcn_readlane((int)v, 63);
}
#define VM0_KNOWN() __builtin_amdgcn_s_waitcnt(0x0F70)

__device__ __forceinline__ void glds16(const void* g, char* l) {
  __builtin_amdgcn_global_load_lds((const __attribute__((address_space(1))) void*)g, (LAS void*)l, 16, 0, 0);
}

__global__ __launch_bounds__(NTHREADS, 3) void a2a_gemm_q8(Params P) {
  __shared__ __attribute__((aligned(16))) char smem[NS * STAGE + 512];
  const int tid = threadIdx.x;
  const int wave = __builtin_amdgcn_readfirstlane(tid >> 6);
  const int lane = tid & 63;
  const int wg = blockIdx.x;
  const int rank = P.rank;
  const unsigned lds0 = (unsigned)(size_t)(LAS char*)smem;
  const unsigned ctrl = lds0 + NS * STAGE;
#define CW(i) (ctrl + 4u * (unsigned)(i))
  char* myws = P.ws[rank];
  const int xg = wg & 7, idx = wg >> 3;
  const int hk = idx & 1;
  const int tm = (idx >> 1) & 3, tn = xg * 4 + (idx >> 3);

  if (wave == 8) {
    const unsigned long long tstart = __builtin_amdgcn_s_memrealtime();
    const bool active = wg < (NR - 1) * PUSHW;
    const int dst = (rank + 1 + wg / PUSHW) & (NR - 1);
    const int pw = wg % PUSHW;
    const int ps = pw >> 2, ptm = pw & 3;
    u32x4 d[16];
    const int r8 = lane >> 3, ch = lane & 7;
    if (active) {
      const unsigned short* src = P.x + (size_t)(dst * MLOC + ptm * 128 + r8) * KSH + ps * 64 + ch * 8;
#pragma unroll
      for (int j = 0; j < 16; ++j) d[j] = *(const u32x4*)(src + (size_t)j * 8 * KSH);
    }
    bar();
    if (!active) return;
    VM0_KNOWN();
    while (__builtin_amdgcn_s_memrealtime() - tstart < (unsigned long long)(ps * PUSH_T)) __builtin_amdgcn_s_sleep(2);
    char* dbase = P.ws[dst] + WS_A + ((size_t)(rank * 8 + ps) * MLOC + ptm * 128 + r8) * 128 + ch * 16;
#pragma unroll
    for (int j = 0; j < 16; ++j) {
      char* a_ = dbase + j * 1024;
      asm volatile("global_store_dwordx4 %0, %1, off sc0 sc1\n\ts_nop 1" :: "v"(a_), "v"(d[j]) : "memory");
      if ((j & 3) == 3) WAITV(0);
    }
    WAITV(0);
    unsigned seq;
    do { seq = (unsigned)__builtin_amdgcn_readfirstlane(lds_rd(CW(C_SEQ))); } while (seq == 0);
    unsigned* fl = (unsigned*)(P.ws[dst] + WS_FLAG) + ptm * 64 + rank * 8 + ps;
    if (lane == 0) asm volatile("global_store_dword %0, %1, off sc0 sc1" :: "v"(fl), "v"(seq) : "memory");
    return;
  }

  if (wave == 9) {
    if (lane < 12) lds_wr(CW(lane), 0);
    WAITL0();
    bar();
    unsigned ticket = 0;
    if (lane == 0)
      ticket = __hip_atomic_fetch_add((unsigned*)(myws + WS_CNT), 1u, __ATOMIC_RELAXED, __HIP_MEMORY_SCOPE_AGENT);
    const unsigned seq = (unsigned)__builtin_amdgcn_readfirstlane(ticket) / NWG + 1u;
    lds_wr(CW(C_SEQ), seq);
    const unsigned* fp = (const unsigned*)(myws + WS_FLAG) + tm * 64 + lane;
    const unsigned long long t0 = __builtin_amdgcn_s_memrealtime();
    unsigned long long seen = (0xFFull << (rank * 8)) | (hk ? 0x5555555555555555ull : 0xAAAAAAAAAAAAAAAAull);
    int cnt = 0;
    unsigned spins = 0;
    while (cnt < NREM) {
      const unsigned mn = __hip_atomic_load(fp, __ATOMIC_RELAXED, __HIP_MEMORY_SCOPE_SYSTEM);
      const unsigned long long rdy = __ballot(mn >= seq);
      const unsigned long long nw = rdy & ~seen;
      if (nw) {
        if ((nw >> lane) & 1ull) {
          const int pos = cnt + __popcll(nw & ((1ull << lane) - 1ull));
          lds_wr(CW(C_QUEUE + pos), (unsigned)lane);
        }
        WAITL0();
        cnt += __popcll(nw);
        seen |= nw;
        lds_wr(CW(C_RDY), (unsigned)cnt);
      } else if ((++spins & 63u) == 0u) {
        if (__builtin_amdgcn_s_memrealtime() - t0 > DEADLINE_TICKS ||
            __hip_atomic_load((unsigned*)(myws + WS_TMO), __ATOMIC_RELAXED, __HIP_MEMORY_SCOPE_AGENT) == seq) {
          if (lane == 0)
            __hip_atomic_store((unsigned*)(myws + WS_TMO), seq, __ATOMIC_RELAXED, __HIP_MEMORY_SCOPE_AGENT);
          lds_wr(CW(C_FAIL), 1);
          if (lane < NREM) lds_wr(CW(C_QUEUE + lane), (unsigned)(rank * 8));
          WAITL0();
          lds_wr(CW(C_RDY), NREM);
          cnt = NREM;
        }
      }
    }
    if (wg < 8) {
      const u64* gp = (const u64*)(myws + WS_AMAX) + wg * 512 + lane;
      bool ok = false;
      unsigned gl = 0, pass = 0;
      for (;; ++pass) {
        if (!ok) {
          ok = true; gl = 0;
#pragma unroll
          for (int k = 0; k < 8; ++k) {
            u64 x = __hip_atomic_load(gp + k * 64, __ATOMIC_RELAXED, __HIP_MEMORY_SCOPE_SYSTEM);
            ok &= (unsigned)(x >> 32) >= seq;
            gl = max(gl, (unsigned)x);
          }
        }
        if (__all(ok)) break;
        if ((pass & 31u) == 31u &&
            (__builtin_amdgcn_s_memrealtime() - t0 > DEADLINE_TICKS ||
             __hip_atomic_load((unsigned*)(myws + WS_TMO), __ATOMIC_RELAXED, __HIP_MEMORY_SCOPE_AGENT) == seq)) {
          if (lane == 0)
            __hip_atomic_store((unsigned*)(myws + WS_TMO), seq, __ATOMIC_RELAXED, __HIP_MEMORY_SCOPE_AGENT);
          gl = 0xFFFFFFFFu;
          break;
        }
      }
      gl = wave_max_u32(gl);
      if (lane < 8)
        __hip_atomic_store((u64*)(myws + WS_FIN + lane * 128) + wg, ((u64)seq << 32) | gl, __ATOMIC_RELAXED, __HIP_MEMORY_SCOPE_AGENT);
    }
    return;
  }

  if (wave >= 4) {
    const int lw = wave - 4;
    const int r8 = lane >> 3;
    const int chA = (lane & 7) ^ r8;
    const unsigned offA_own = (unsigned)(tm * BM + lw * 32 + r8) * 1024u + chA * 16;
    const unsigned offA_stg = (unsigned)(tm * BM + lw * 32 + r8) * 128u + chA * 16;
    unsigned offB[4];
#pragma unroll
    for (int j = 0; j < 4; ++j) {
      const int k = (lw * 4 + j) * 4 + (lane >> 4);
      const int f = (k & 3) | (((k >> 3) & 1) << 2);
      const int sl = lane & 15;
      const int ch = (((sl >> 1) ^ f) << 1) | (sl & 1);
      offB[j] = (unsigned)k * (NDIM * 2) + tn * (BN * 2) + ch * 16;
    }
    const char* xown = (const char*)P.x + (size_t)rank * MLOC * 1024;
    const char* wsA = myws + WS_A;
    const char* wbase = (const char*)P.w;
    int li = 0, pub = 0, slot = 0, avail = NOWN, mind = 0, unext = 0;
    while (li < NSTEP) {
      while (li >= avail || li >= mind + NS) {
        if (li - pub == 2) { WAITV(8); ++pub; lds_wr(CW(C_FULL + lw), (unsigned)pub); }
        else if (li - pub == 1) { WAITV(0); ++pub; lds_wr(CW(C_FULL + lw), (unsigned)pub); }
        mind = lds_rd_min4(CW(C_DONE));
        avail = NOWN + __builtin_amdgcn_readfirstlane((int)lds_rd(CW(C_RDY)));
        unext = __builtin_amdgcn_readfirstlane((int)lds_rd(CW(C_QUEUE + (li >= NOWN ? li - NOWN : 0))));
      }
      int p, s;
      const char* abase;
      unsigned aoffs, astep;
      if (li < NOWN) {
        p = rank; s = 2 * li + hk;
        abase = xown + s * 128; aoffs = offA_own; astep = 8 * 1024;
      } else {
        p = unext >> 3; s = unext & 7;
        abase = wsA + (size_t)unext * 65536; aoffs = offA_stg; astep = 1024;
      }
      char* sb = smem + slot * STAGE;
#pragma unroll
      for (int j = 0; j < 4; ++j)
        glds16(abase + aoffs + j * astep, sb + (lw * 4 + j) * 1024 + lane * 16);
      const char* bb = wbase + (size_t)(p * KSH + s * 64) * (NDIM * 2);
#pragma unroll
      for (int j = 0; j < 4; ++j)
        glds16(bb + offB[j], sb + A_BYTES + (lw * 4 + j) * 1024 + lane * 16);
      ++li;
      slot = (slot == NS - 1) ? 0 : slot + 1;
      if (li == 1) bar();
      u32x4 vd; unsigned vr, vq;
      asm volatile("ds_read_b128 %0, %3\n\tds_read_b32 %1, %4\n\tds_read_b32 %2, %5"
                   : "=&v"(vd), "=&v"(vr), "=&v"(vq)
                   : "v"(CW(C_DONE)), "v"(CW(C_RDY)), "v"(CW(C_QUEUE + (li >= NOWN ? li - NOWN : 0))) : "memory");
      if (li - pub == 3) { WAITV(16); ++pub; lds_wr(CW(C_FULL + lw), (unsigned)pub); }
      asm volatile("s_waitcnt lgkmcnt(0)" : "+v"(vd), "+v"(vr), "+v"(vq) :: "memory");
      mind = __builtin_amdgcn_readfirstlane((int)min(min(vd[0], vd[1]), min(vd[2], vd[3])));
      avail = NOWN + __builtin_amdgcn_readfirstlane((int)vr);
      unext = __builtin_amdgcn_readfirstlane((int)vq);
    }
    WAITV(8); lds_wr(CW(C_FULL + lw), (unsigned)(NSTEP - 1));
    WAITV(0); lds_wr(CW(C_FULL + lw), (unsigned)NSTEP);
    return;
  }

  const int wm = wave >> 1, wn = wave & 1;
  const int fr = lane & 15, fq = lane >> 4;
  unsigned aoff[2];
#pragma unroll
  for (int kk = 0; kk < 2; ++kk)
    aoff[kk] = lds0 + (unsigned)(wm * 64 + fr) * 128u + (((kk * 4 + fq) ^ (lane & 7)) * 16);
  unsigned boff[4];
  {
    const int q = (lane & 15) >> 2, p = lane & 3, g = fq;
    const int f = q | ((g & 1) << 2);
#pragma unroll
    for (int nt = 0; nt < 4; ++nt)
      boff[nt] = lds0 + A_BYTES + (unsigned)(8 * g + q) * 256u + (((wn * 4 + nt) ^ f) * 32) + p * 8;
  }

  u64* gdst = (u64*)(P.ws[lane & (NR - 1)] + WS_AMAX) + (rank * NWG + wg) * 2 + wn;
  f32x4 acc[4][4];
#pragma unroll
  for (int i = 0; i < 4; ++i)
#pragma unroll
    for (int j = 0; j < 4; ++j) acc[i][j] = f32x4{0.f, 0.f, 0.f, 0.f};

  bar();
  if (wave == 0) STAMP(0);
  {
    bf16x8 a[4][2];
    s16x4 bl[4][2], bh[4][2];
#define RD_K0(sb_) asm volatile( \
          "ds_read_b64_tr_b16 %0, %12\n\t" \
          "ds_read_b64_tr_b16 %1, %12 offset:1024\n\t" \
          "ds_read_b64_tr_b16 %2, %13\n\t" \
          "ds_read_b64_tr_b16 %3, %13 offset:1024\n\t" \
          "ds_read_b128 %8, %16\n\t" \
          "ds_read_b128 %9, %16 offset:2048\n\t" \
          "ds_read_b128 %10, %16 offset:4096\n\t" \
          "ds_read_b128 %11, %16 offset:6144\n\t" \
          "ds_read_b64_tr_b16 %4, %14\n\t" \
          "ds_read_b64_tr_b16 %5, %14 offset:1024\n\t" \
          "ds_read_b64_tr_b16 %6, %15\n\t" \
          "ds_read_b64_tr_b16 %7, %15 offset:1024" \
          : "=&v"(bl[0][0]), "=&v"(bh[0][0]), "=&v"(bl[1][0]), "=&v"(bh[1][0]), \
            "=&v"(bl[2][0]), "=&v"(bh[2][0]), "=&v"(bl[3][0]), "=&v"(bh[3][0]), \
            "=&v"(a[0][0]), "=&v"(a[1][0]), "=&v"(a[2][0]), "=&v"(a[3][0]) \
          : "v"((sb_) + boff[0]), "v"((sb_) + boff[1]), "v"((sb_) + boff[2]), "v"((sb_) + boff[3]), "v"((sb_) + aoff[0]) : "memory")
#define RD_K1(sb_) asm volatile( \
          "ds_read_b64_tr_b16 %0, %12 offset:8192\n\t" \
          "ds_read_b64_tr_b16 %1, %12 offset:9216\n\t" \
          "ds_read_b64_tr_b16 %2, %13 offset:8192\n\t" \
          "ds_read_b64_tr_b16 %3, %13 offset:9216\n\t" \
          "ds_read_b128 %8, %16\n\t" \
          "ds_read_b128 %9, %16 offset:2048\n\t" \
          "ds_read_b128 %10, %16 offset:4096\n\t" \
          "ds_read_b128 %11, %16 offset:6144\n\t" \
          "ds_read_b64_tr_b16 %4, %14 offset:8192\n\t" \
          "ds_read_b64_tr_b16 %5, %14 offset:9216\n\t" \
          "ds_read_b64_tr_b16 %6, %15 offset:8192\n\t" \
          "ds_read_b64_tr_b16 %7, %15 offset:9216" \
          : "=&v"(bl[0][1]), "=&v"(bh[0][1]), "=&v"(bl[1][1]), "=&v"(bh[1][1]), \
            "=&v"(bl[2][1]), "=&v"(bh[2][1]), "=&v"(bl[3][1]), "=&v"(bh[3][1]), \
            "=&v"(a[0][1]), "=&v"(a[1][1]), "=&v"(a[2][1]), "=&v"(a[3][1]) \
          : "v"((sb_) + boff[0]), "v"((sb_) + boff[1]), "v"((sb_) + boff[2]), "v"((sb_) + boff[3]), "v"((sb_) + aoff[1]) : "memory")
#define WAIT_K(kk, n) asm volatile("s_waitcnt lgkmcnt(" #n ")" \
                   : "+v"(bl[0][kk]), "+v"(bh[0][kk]), "+v"(bl[1][kk]), "+v"(bh[1][kk]), \
                     "+v"(bl[2][kk]), "+v"(bh[2][kk]), "+v"(bl[3][kk]), "+v"(bh[3][kk]), \
                     "+v"(a[0][kk]), "+v"(a[1][kk]), "+v"(a[2][kk]), "+v"(a[3][kk]) :: "memory")
#define BFRAG(nt, kk) bf16x8{bl[nt][kk][0], bl[nt][kk][1], bl[nt][kk][2], bl[nt][kk][3], \
                             bh[nt][kk][0], bh[nt][kk][1], bh[nt][kk][2], bh[nt][kk][3]}
#define MMA(nt, kk) do { const bf16x8 bb_ = BFRAG(nt, kk); _Pragma("unroll") for (int mt = 0; mt < 4; ++mt) \
        acc[mt][nt] = __builtin_amdgcn_mfma_f32_16x16x32_bf16(bb_, a[mt][kk], acc[mt][nt], 0, 0, 0); } while (0)
    int minf = 0;
    unsigned sb = 0;
    while (minf <= 0) minf = lds_rd_min4(CW(C_FULL));
    RD_K0(sb);
    RD_K1(sb);
    for (int ci = 0; ci < NSTEP; ++ci) {
      WAIT_K(0, 12);
      __builtin_amdgcn_sched_barrier(0);
      MMA(0, 0); MMA(1, 0); MMA(2, 0); MMA(3, 0);
      __builtin_amdgcn_sched_barrier(0);
      WAIT_K(1, 0);
      lds_wr(CW(C_DONE + wave), (unsigned)(ci + 1));
      sb = (sb == (NS - 1) * STAGE) ? 0u : sb + STAGE;
      const bool more = ci + 1 < NSTEP;
      if (more) {
        while (minf <= ci + 1) minf = lds_rd_min4(CW(C_FULL));
        RD_K0(sb);
      }
      __builtin_amdgcn_sched_barrier(0);
      MMA(0, 1); MMA(1, 1); MMA(2, 1); MMA(3, 1);
      __builtin_amdgcn_sched_barrier(0);
      if (more) RD_K1(sb);
    }
  }

  if (wave == 0) STAMP(2);
  const unsigned seq = (unsigned)__builtin_amdgcn_readfirstlane(lds_rd(CW(C_SEQ)));
  bool failed = lds_rd(CW(C_FAIL)) != 0;
  if (wm != hk) {
    if (wn == 0) STAMP(6);
    char* pb = myws + WS_PART + ((size_t)(wg * 2 + wn) * 8) * 1024 + lane * 16;
#pragma unroll
    for (int i = 0; i < 4; ++i)
#pragma unroll
      for (int j = 0; j < 2; ++j) {
        char* a_ = pb + (i * 2 + j) * 1024;
        u32x4 v;
        asm volatile("v_cvt_pk_bf16_f32 %0, %1, %2" : "=v"(v[0]) : "v"(acc[i][2 * j][0]), "v"(acc[i][2 * j][1]));
        asm volatile("v_cvt_pk_bf16_f32 %0, %1, %2" : "=v"(v[1]) : "v"(acc[i][2 * j][2]), "v"(acc[i][2 * j][3]));
        asm volatile("v_cvt_pk_bf16_f32 %0, %1, %2" : "=v"(v[2]) : "v"(acc[i][2 * j + 1][0]), "v"(acc[i][2 * j + 1][1]));
        asm volatile("v_cvt_pk_bf16_f32 %0, %1, %2" : "=v"(v[3]) : "v"(acc[i][2 * j + 1][2]), "v"(acc[i][2 * j + 1][3]));
        asm volatile("global_store_dwordx4 %0, %1, off sc1\n\ts_nop 1" :: "v"(a_), "v"(v) : "memory");
      }
    WAITV(0);
    if (lane == 0)
      __hip_atomic_store((unsigned*)(myws + WS_XFLAG + (size_t)(wg * 2 + wn) * 128), seq, __ATOMIC_RELAXED, __HIP_MEMORY_SCOPE_AGENT);
    if (wn == 0) STAMP(7);
    return;
  }
  {
    const int pwg = wg ^ 8;
    const unsigned* xf = (const unsigned*)(myws + WS_XFLAG + (size_t)(pwg * 2 + wn) * 128);
    const unsigned long long t0 = __builtin_amdgcn_s_memrealtime();
    unsigned spins = 0;
    while (!failed && __hip_atomic_load(xf, __ATOMIC_RELAXED, __HIP_MEMORY_SCOPE_AGENT) < seq) {
      if ((++spins & 63u) != 0u) continue;
      if (__builtin_amdgcn_s_memrealtime() - t0 > DEADLINE_TICKS ||
          __hip_atomic_load((unsigned*)(myws + WS_TMO), __ATOMIC_RELAXED, __HIP_MEMORY_SCOPE_AGENT) == seq) {
        __hip_atomic_store((unsigned*)(myws + WS_TMO), seq, __ATOMIC_RELAXED, __HIP_MEMORY_SCOPE_AGENT);
        failed = true;
      }
    }
    if (wn == 0) STAMP(15);
    const char* pb = myws + WS_PART + ((size_t)(pwg * 2 + wn) * 8) * 1024 + lane * 16;
    u32x4 t[8];
#pragma unroll
    for (int i = 0; i < 8; ++i) {
      const char* a_ = pb + i * 1024;
      asm volatile("global_load_dwordx4 %0, %1, off sc1" : "=&v"(t[i]) : "v"(a_) : "memory");
    }
    asm volatile("s_waitcnt vmcnt(0)"
                 : "+v"(t[0]), "+v"(t[1]), "+v"(t[2]), "+v"(t[3]), "+v"(t[4]), "+v"(t[5]), "+v"(t[6]), "+v"(t[7]) :: "memory");
#pragma unroll
    for (int i = 0; i < 4; ++i)
#pragma unroll
      for (int j = 0; j < 4; ++j) {
        const unsigned lo = t[i * 2 + (j >> 1)][(j & 1) * 2], hi = t[i * 2 + (j >> 1)][(j & 1) * 2 + 1];
        acc[i][j][0] += __uint_as_float(lo << 16);
        acc[i][j][1] += __uint_as_float(lo & 0xFFFF0000u);
        acc[i][j][2] += __uint_as_float(hi << 16);
        acc[i][j][3] += __uint_as_float(hi & 0xFFFF0000u);
      }
  }
  if (wn == 0) STAMP(1);

  float m = 0.f;
#pragma unroll
  for (int i = 0; i < 4; ++i)
#pragma unroll
    for (int j = 0; j < 4; ++j)
#pragma unroll
      for (int r = 0; r < 4; ++r) m = fmaxf(m, fabsf(acc[i][j][r]));
  {
    const unsigned mw = wave_max_u32(__float_as_uint(m));
    if (lane < NR)
      __hip_atomic_store(gdst, ((u64)seq << 32) | mw, __ATOMIC_RELAXED, __HIP_MEMORY_SCOPE_SYSTEM);
  }
  if (wn == 0) STAMP(3);
  unsigned gm = 0;
  {
    const u64* fp = (const u64*)(myws + WS_FIN + (wg & 7) * 128) + (lane & 7);
    const unsigned long long t0 = __builtin_amdgcn_s_memrealtime();
    unsigned spins = 0;
    while (!failed) {
      const u64 x = __hip_atomic_load(fp, __ATOMIC_RELAXED, __HIP_MEMORY_SCOPE_AGENT);
      if (__all((unsigned)(x >> 32) >= seq)) { gm = wave_max_u32((unsigned)x); failed = gm == 0xFFFFFFFFu; break; }
      __builtin_amdgcn_s_sleep(2);
      if ((++spins & 63u) != 0u) continue;
      if (__builtin_amdgcn_s_memrealtime() - t0 > DEADLINE_TICKS ||
          __hip_atomic_load((unsigned*)(myws + WS_TMO), __ATOMIC_RELAXED, __HIP_MEMORY_SCOPE_AGENT) == seq) {
        __hip_atomic_store((unsigned*)(myws + WS_TMO), seq, __ATOMIC_RELAXED, __HIP_MEMORY_SCOPE_AGENT);
        failed = true;
      }
    }
  }
  const float amax = __uint_as_float(gm);
  if (wn == 0) STAMP(4);
  float scale = amax / 127.0f;
  if (!(scale > 0.f)) scale = 1.0f;
  const float rscale = 1.0f / scale;

  signed char* tile = (signed char*)smem + wn * 4096;
#pragma unroll
  for (int mt = 0; mt < 4; ++mt)
#pragma unroll
    for (int nt = 0; nt < 4; ++nt) {
      unsigned pk = 0;
#pragma unroll
      for (int r = 0; r < 4; ++r) {
        const float y = acc[mt][nt][r];
        float q0 = y * rscale;
        q0 = fmaf(fmaf(-q0, scale, y), rscale, q0);
        float qv = rintf(q0);
        qv = fminf(fmaxf(qv, -127.f), 127.f);
        int qi = failed ? -128 : (int)qv;
        pk |= ((unsigned)qi & 0xFFu) << (8 * r);
      }
      *(unsigned*)(tile + (mt * 16 + fr) * 64 + nt * 16 + fq * 4) = pk;
    }
  WAITL0();
  __builtin_amdgcn_wave_barrier();
#pragma unroll
  for (int j = 0; j < 4; ++j) {
    const int c = lane + j * 64;
    const int row = c >> 2, cc = c & 3;
    u32x4 v = *(const u32x4*)((const char*)tile + row * 64 + cc * 16);
    *(u32x4*)(P.out + (size_t)(tm * BM + hk * 64 + row) * NDIM + tn * BN + wn * 64 + cc * 16) = v;
  }
  if (wn == 0) STAMP(5);
}

}

extern "C" void kernel_launch(int rank, int nranks, ncclComm_t comm,
                              hipStream_t stream,
                              void* const* d_in, const int* in_sizes, int n_in,
                              void* d_out, int out_size,
                              const int* in_dtypes, int out_dtype,
                              void* const* d_out_all,
                              void* const* d_ws_all, size_t ws_bytes) {
  (void)comm; (void)in_sizes; (void)n_in; (void)out_size; (void)in_dtypes; (void)out_dtype; (void)d_out_all;
  if (nranks != NR || ws_bytes < WS_NEED || (unsigned)rank >= (unsigned)NR) {
    fprintf(stderr, "unexpected configuration\n");
    return;
  }
  Params P;
  P.x = (const unsigned short*)d_in[0];
  P.w = (const unsigned short*)d_in[1];
  P.out = (signed char*)d_out;
  for (int p = 0; p < NR; ++p) P.ws[p] = (char*)d_ws_all[p];
  P.rank = rank;
  hipLaunchKernelGGL(a2a_gemm_q8, dim3(NWG), dim3(NTHREADS), 0, stream, P);
  HIP_CHECK(hipGetLastError());
}
```

```cpp
#include <hip/hip_runtime.h>
#include <rccl/rccl.h>
#include <cstdio>
#include <cstdint>

#define HIP_CHECK(expr) do { hipError_t _e = (expr); if (_e != hipSuccess) { \
    fprintf(stderr, "HIP error %s at %s:%d\n", hipGetErrorString(_e), __FILE__, __LINE__); \
    return; } } while (0)

typedef __attribute__((ext_vector_type(8))) short bf16x8;
typedef __attribute__((ext_vector_type(4))) short s16x4;
typedef __attribute__((ext_vector_type(4))) float f32x4;
typedef __attribute__((ext_vector_type(4))) unsigned u32x4;
typedef unsigned long long u64;

#define LAS __attribute__((address_space(3)))

namespace {

constexpr int NR = 8;
constexpr int MLOC = 512;
constexpr int KSH = 512;
constexpr int NDIM = 4096;
constexpr int BM = 128, BN = 128, BK = 64;
constexpr int NS = 4;
constexpr int A_BYTES = BM * BK * 2;
constexpr int B_BYTES = BK * BN * 2;
constexpr int STAGE = A_BYTES + B_BYTES;
constexpr int NWG = 256;
constexpr int NSTEP = 32;
constexpr int NOWN = 4;
constexpr int NREM = NSTEP - NOWN;
constexpr int PUSHW = 32;
constexpr int NTHREADS = 640;
#define PACE_N 4
#define PACE_FROM 2
#define PUSH_T 170

constexpr size_t WS_A = 0;
constexpr size_t WS_FLAG = 4u << 20;
constexpr size_t WS_AMAX = (4u << 20) + (64u << 10);
constexpr size_t WS_CNT = (4u << 20) + (128u << 10);
constexpr size_t WS_TMO = WS_CNT + 4096;
constexpr size_t WS_FIN = (4u << 20) + (160u << 10);
constexpr size_t WS_XFLAG = (4u << 20) + (192u << 10);
constexpr size_t WS_PART = (6u << 20);
constexpr size_t WS_NEED = WS_PART + (8u << 20);
#define STAMP(i) do { } while (0)

constexpr unsigned long long DEADLINE_TICKS = 40000000ull;

constexpr int C_FULL = 0;
constexpr int C_DONE = 4;
constexpr int C_RDY = 8;
constexpr int C_SEQ = 9;
constexpr int C_FAIL = 10;
constexpr int C_QUEUE = 32;

struct Params {
  const unsigned short* x;
  const unsigned short* w;
  signed char* out;
  char* ws[NR];
  int rank;
};

__device__ __forceinline__ void bar() { __builtin_amdgcn_s_barrier(); }
#define WAITV(n) asm volatile("s_waitcnt vmcnt(" #n ")" ::: "memory")
#define WAITL0() asm volatile("s_waitcnt lgkmcnt(0)" ::: "memory")

__device__ __forceinline__ void lds_wr(unsigned addr, unsigned v) {
  asm volatile("ds_write_b32 %0, %1" :: "v"(addr), "v"(v) : "memory");
}
__device__ __forceinline__ unsigned lds_rd(unsigned addr) {
  unsigned v;
  asm volatile("ds_read_b32 %0, %1\n\ts_waitcnt lgkmcnt(0)" : "=&v"(v) : "v"(addr) : "memory");
  return v;
}
__device__ __forceinline__ int lds_rd_min4(unsigned addr) {
  u32x4 v;
  asm volatile("ds_read_b128 %0, %1\n\ts_waitcnt lgkmcnt(0)" : "=&v"(v) : "v"(addr) : "memory");
  unsigned m = min(min(v[0], v[1]), min(v[2], v[3]));
  return __builtin_amdgcn_readfirstlane((int)m);
}
__device__ __forceinline__ unsigned wave_max_u32(unsigned v) {
  v = max(v, (unsigned)__builtin_amdgcn_update_dpp(0, (int)v, 0x111, 0xF, 0xF, false));
  v = max(v, (unsigned)__builtin_amdgcn_update_dpp(0, (int)v, 0x112, 0xF, 0xF, false));
  v = max(v, (unsigned)__builtin_amdgcn_update_dpp(0, (int)v, 0x114, 0xF, 0xF, false));
  v = max(v, (unsigned)__builtin_amdgcn_update_dpp(0, (int)v, 0x118, 0xF, 0xF, false));
  v = max(v, (unsigned)__builtin_amdgcn_update_dpp(0, (int)v, 0x142, 0xA, 0xF, false));
  v = max(v, (unsigned)__builtin_amdgcn_update_dpp(0, (int)v, 0x143, 0xC, 0xF, false));
  return (unsigned)__builtin_amdgcn_readlane((int)v, 63);
}
#define VM0_KNOWN() __builtin_amdgcn_s_waitcnt(0x0F70)

__device__ __forceinline__ void glds16(const void* g, char* l) {
  __builtin_amdgcn_global_load_lds((const __attribute__((address_space(1))) void*)g, (LAS void*)l, 16, 0, 0);
}

__global__ __launch_bounds__(NTHREADS, 3) void a2a_gemm_q8(Params P) {
  __shared__ __attribute__((aligned(16))) char smem[NS * STAGE + 512];
  const int tid = threadIdx.x;
  const int wave = __builtin_amdgcn_readfirstlane(tid >> 6);
  const int lane = tid & 63;
  const int wg = blockIdx.x;
  const int rank = P.rank;
  const unsigned lds0 = (unsigned)(size_t)(LAS char*)smem;
  const unsigned ctrl = lds0 + NS * STAGE;
#define CW(i) (ctrl + 4u * (unsigned)(i))
  char* myws = P.ws[rank];
  const int xg = wg & 7, idx = wg >> 3;
  const int hk = idx & 1;
  const int tm = (idx >> 1) & 3, tn = xg * 4 + (idx >> 3);

  if (wave == 8) {
    const unsigned long long tstart = __builtin_amdgcn_s_memrealtime();
    const bool active = wg < (NR - 1) * PUSHW;
    const int dst = (rank + 1 + wg / PUSHW) & (NR - 1);
    const int pw = wg % PUSHW;
    const int ps = pw >> 2, ptm = pw & 3;
    u32x4 d[16];
    const int r8 = lane >> 3, ch = lane & 7;
    if (active) {
      const unsigned short* src = P.x + (size_t)(dst * MLOC + ptm * 128 + r8) * KSH + ps * 64 + ch * 8;
#pragma unroll
      for (int j = 0; j < 16; ++j) d[j] = *(const u32x4*)(src + (size_t)j * 8 * KSH);
    }
    bar();
    if (!active) return;
    VM0_KNOWN();
    while (__builtin_amdgcn_s_memrealtime() - tstart < (unsigned long long)(ps * PUSH_T)) __builtin_amdgcn_s_sleep(2);
    char* dbase = P.ws[dst] + WS_A + ((size_t)(rank * 8 + ps) * MLOC + ptm * 128 + r8) * 128 + ch * 16;
#pragma unroll
    for (int j = 0; j < 16; ++j) {
      char* a_ = dbase + j * 1024;
      asm volatile("global_store_dwordx4 %0, %1, off sc0 sc1\n\ts_nop 1" :: "v"(a_), "v"(d[j]) : "memory");
      if ((j % PACE_N) == PACE_N - 1 && ps >= PACE_FROM) WAITV(0);
    }
    WAITV(0);
    unsigned seq;
    do { seq = (unsigned)__builtin_amdgcn_readfirstlane(lds_rd(CW(C_SEQ))); } while (seq == 0);
    unsigned* fl = (unsigned*)(P.ws[dst] + WS_FLAG) + ptm * 64 + rank * 8 + ps;
    if (lane == 0) asm volatile("global_store_dword %0, %1, off sc0 sc1" :: "v"(fl), "v"(seq) : "memory");
    return;
  }

  if (wave == 9) {
    if (lane < 12) lds_wr(CW(lane), 0);
    WAITL0();
    bar();
    unsigned ticket = 0;
    if (lane == 0)
      ticket = __hip_atomic_fetch_add((unsigned*)(myws + WS_CNT + (wg & 15) * 128), 1u, __ATOMIC_RELAXED, __HIP_MEMORY_SCOPE_AGENT);
    const unsigned seq = (unsigned)__builtin_amdgcn_readfirstlane(ticket) / (NWG / 16) + 1u;
    lds_wr(CW(C_SEQ), seq);
    const unsigned* fp = (const unsigned*)(myws + WS_FLAG) + tm * 64 + lane;
    const unsigned long long t0 = __builtin_amdgcn_s_memrealtime();
    unsigned long long seen = (0xFFull << (rank * 8)) | (hk ? 0x5555555555555555ull : 0xAAAAAAAAAAAAAAAAull);
    int cnt = 0;
    unsigned spins = 0;
    while (cnt < NREM) {
      const unsigned mn = __hip_atomic_load(fp, __ATOMIC_RELAXED, __HIP_MEMORY_SCOPE_SYSTEM);
      const unsigned long long rdy = __ballot(mn >= seq);
      const unsigned long long nw = rdy & ~seen;
      if (nw) {
        if ((nw >> lane) & 1ull) {
          const int pos = cnt + __popcll(nw & ((1ull << lane) - 1ull));
          lds_wr(CW(C_QUEUE + pos), (unsigned)lane);
        }
        WAITL0();
        cnt += __popcll(nw);
        seen |= nw;
        lds_wr(CW(C_RDY), (unsigned)cnt);
      } else if ((++spins & 63u) == 0u) {
        if (__builtin_amdgcn_s_memrealtime() - t0 > DEADLINE_TICKS ||
            __hip_atomic_load((unsigned*)(myws + WS_TMO), __ATOMIC_RELAXED, __HIP_MEMORY_SCOPE_AGENT) == seq) {
          if (lane == 0)
            __hip_atomic_store((unsigned*)(myws + WS_TMO), seq, __ATOMIC_RELAXED, __HIP_MEMORY_SCOPE_AGENT);
          lds_wr(CW(C_FAIL), 1);
          if (lane < NREM) lds_wr(CW(C_QUEUE + lane), (unsigned)(rank * 8));
          WAITL0();
          lds_wr(CW(C_RDY), NREM);
          cnt = NREM;
        }
      }
    }
    if (wg < 8) {
      const u64* gp = (const u64*)(myws + WS_AMAX) + wg * 512 + lane;
      bool ok = false;
      unsigned gl = 0, pass = 0;
      for (;; ++pass) {
        if (!ok) {
          ok = true; gl = 0;
#pragma unroll
          for (int k = 0; k < 8; ++k) {
            u64 x = __hip_atomic_load(gp + k * 64, __ATOMIC_RELAXED, __HIP_MEMORY_SCOPE_SYSTEM);
            ok &= (unsigned)(x >> 32) >= seq;
            gl = max(gl, (unsigned)x);
          }
        }
        if (__all(ok)) break;
        if ((pass & 31u) == 31u &&
            (__builtin_amdgcn_s_memrealtime() - t0 > DEADLINE_TICKS ||
             __hip_atomic_load((unsigned*)(myws + WS_TMO), __ATOMIC_RELAXED, __HIP_MEMORY_SCOPE_AGENT) == seq)) {
          if (lane == 0)
            __hip_atomic_store((unsigned*)(myws + WS_TMO), seq, __ATOMIC_RELAXED, __HIP_MEMORY_SCOPE_AGENT);
          gl = 0xFFFFFFFFu;
          break;
        }
      }
      gl = wave_max_u32(gl);
      if (lane < 8)
        __hip_atomic_store((u64*)(myws + WS_FIN + lane * 128) + wg, ((u64)seq << 32) | gl, __ATOMIC_RELAXED, __HIP_MEMORY_SCOPE_AGENT);
    }
    return;
  }

  if (wave >= 4) {
    const int lw = wave - 4;
    const int r8 = lane >> 3;
    const int chA = (lane & 7) ^ r8;
    const unsigned offA_own = (unsigned)(tm * BM + lw * 32 + r8) * 1024u + chA * 16;
    const unsigned offA_stg = (unsigned)(tm * BM + lw * 32 + r8) * 128u + chA * 16;
    unsigned offB[4];
#pragma unroll
    for (int j = 0; j < 4; ++j) {
      const int k = (lw * 4 + j) * 4 + (lane >> 4);
      const int f = (k & 3) | (((k >> 3) & 1) << 2);
      const int sl = lane & 15;
      const int ch = (((sl >> 1) ^ f) << 1) | (sl & 1);
      offB[j] = (unsigned)k * (NDIM * 2) + tn * (BN * 2) + ch * 16;
    }
    const char* xown = (const char*)P.x + (size_t)rank * MLOC * 1024;
    const char* wsA = myws + WS_A;
    const char* wbase = (const char*)P.w;
    int li = 0, pub = 0, slot = 0, avail = NOWN, mind = 0, unext = 0;
    while (li < NSTEP) {
      while (li >= avail || li >= mind + NS) {
        if (li - pub == 2) { WAITV(8); ++pub; lds_wr(CW(C_FULL + lw), (unsigned)pub); }
        else if (li - pub == 1) { WAITV(0); ++pub; lds_wr(CW(C_FULL + lw), (unsigned)pub); }
        mind = lds_rd_min4(CW(C_DONE));
        avail = NOWN + __builtin_amdgcn_readfirstlane((int)lds_rd(CW(C_RDY)));
        unext = __builtin_amdgcn_readfirstlane((int)lds_rd(CW(C_QUEUE + (li >= NOWN ? li - NOWN : 0))));
      }
      int p, s;
      const char* abase;
      unsigned aoffs, astep;
      if (li < NOWN) {
        p = rank; s = 2 * li + hk;
        abase = xown + s * 128; aoffs = offA_own; astep = 8 * 1024;
      } else {
        p = unext >> 3; s = unext & 7;
        abase = wsA + (size_t)unext * 65536; aoffs = offA_stg; astep = 1024;
      }
      char* sb = smem + slot * STAGE;
#pragma unroll
      for (int j = 0; j < 4; ++j)
        glds16(abase + aoffs + j * astep, sb + (lw * 4 + j) * 1024 + lane * 16);
      const char* bb = wbase + (size_t)(p * KSH + s * 64) * (NDIM * 2);
#pragma unroll
      for (int j = 0; j < 4; ++j)
        glds16(bb + offB[j], sb + A_BYTES + (lw * 4 + j) * 1024 + lane * 16);
      ++li;
      slot = (slot == NS - 1) ? 0 : slot + 1;
      if (li == 1) bar();
      u32x4 vd; unsigned vr, vq;
      asm volatile("ds_read_b128 %0, %3\n\tds_read_b32 %1, %4\n\tds_read_b32 %2, %5"
                   : "=&v"(vd), "=&v"(vr), "=&v"(vq)
                   : "v"(CW(C_DONE)), "v"(CW(C_RDY)), "v"(CW(C_QUEUE + (li >= NOWN ? li - NOWN : 0))) : "memory");
      if (li - pub == 3) { WAITV(16); ++pub; lds_wr(CW(C_FULL + lw), (unsigned)pub); }
      asm volatile("s_waitcnt lgkmcnt(0)" : "+v"(vd), "+v"(vr), "+v"(vq) :: "memory");
      mind = __builtin_amdgcn_readfirstlane((int)min(min(vd[0], vd[1]), min(vd[2], vd[3])));
      avail = NOWN + __builtin_amdgcn_readfirstlane((int)vr);
      unext = __builtin_amdgcn_readfirstlane((int)vq);
    }
    WAITV(8); lds_wr(CW(C_FULL + lw), (unsigned)(NSTEP - 1));
    WAITV(0); lds_wr(CW(C_FULL + lw), (unsigned)NSTEP);
    return;
  }

  const int wm = wave >> 1, wn = wave & 1;
  const int fr = lane & 15, fq = lane >> 4;
  unsigned aoff[2];
#pragma unroll
  for (int kk = 0; kk < 2; ++kk)
    aoff[kk] = lds0 + (unsigned)(wm * 64 + fr) * 128u + (((kk * 4 + fq) ^ (lane & 7)) * 16);
  unsigned boff[4];
  {
    const int q = (lane & 15) >> 2, p = lane & 3, g = fq;
    const int f = q | ((g & 1) << 2);
#pragma unroll
    for (int nt = 0; nt < 4; ++nt)
      boff[nt] = lds0 + A_BYTES + (unsigned)(8 * g + q) * 256u + (((wn * 4 + nt) ^ f) * 32) + p * 8;
  }

  u64* gdst = (u64*)(P.ws[lane & (NR - 1)] + WS_AMAX) + (rank * NWG + wg) * 2 + wn;
  f32x4 acc[4][4];
#pragma unroll
  for (int i = 0; i < 4; ++i)
#pragma unroll
    for (int j = 0; j < 4; ++j) acc[i][j] = f32x4{0.f, 0.f, 0.f, 0.f};

  bar();
  if (wave == 0) STAMP(0);
  {
    bf16x8 a[4][2];
    s16x4 bl[4][2], bh[4][2];
#define WAIT_K(kk, n) asm volatile("s_waitcnt lgkmcnt(" #n ")" \
                   : "+v"(bl[0][kk]), "+v"(bh[0][kk]), "+v"(bl[1][kk]), "+v"(bh[1][kk]), \
                     "+v"(bl[2][kk]), "+v"(bh[2][kk]), "+v"(bl[3][kk]), "+v"(bh[3][kk]), \
                     "+v"(a[0][kk]), "+v"(a[1][kk]), "+v"(a[2][kk]), "+v"(a[3][kk]) :: "memory")
#define BFRAG(nt, kk) bf16x8{bl[nt][kk][0], bl[nt][kk][1], bl[nt][kk][2], bl[nt][kk][3], \
                             bh[nt][kk][0], bh[nt][kk][1], bh[nt][kk][2], bh[nt][kk][3]}
#define MMA(nt, kk) do { const bf16x8 bb_ = BFRAG(nt, kk); _Pragma("unroll") for (int mt = 0; mt < 4; ++mt) \
        acc[mt][nt] = __builtin_amdgcn_mfma_f32_16x16x32_bf16(bb_, a[mt][kk], acc[mt][nt], 0, 0, 0); } while (0)
    typedef short v4i16_t __attribute__((ext_vector_type(4)));
#define LDT(addr) __builtin_bit_cast(s16x4, __builtin_amdgcn_ds_read_tr16_b64_v4i16((LAS v4i16_t*)(size_t)(addr)))
#define LDA(addr) (*(const LAS bf16x8*)(size_t)(addr))
#define RDB(kk, sb_) do { _Pragma("unroll") for (int nt = 0; nt < 4; ++nt) { \
        bl[nt][kk] = LDT((sb_) + boff[nt] + (kk) * 8192); bh[nt][kk] = LDT((sb_) + boff[nt] + (kk) * 8192 + 1024); } \
        _Pragma("unroll") for (int mt = 0; mt < 4; ++mt) a[mt][kk] = LDA((sb_) + aoff[kk] + mt * 2048); } while (0)
#define ILV_PATTERN() do { _Pragma("unroll") for (int q = 0; q < 12; ++q) { \
        __builtin_amdgcn_sched_group_barrier(0x8, 1, 0); __builtin_amdgcn_sched_group_barrier(0x100, 1, 0); } \
        __builtin_amdgcn_sched_group_barrier(0x8, 4, 0); } while (0)
    int minf = 0;
    unsigned sb = 0;
    while (minf <= 0) minf = lds_rd_min4(CW(C_FULL));
    RDB(0, sb);
    __builtin_amdgcn_sched_barrier(0);
    for (int ci = 0; ci < NSTEP; ++ci) {
      RDB(1, sb);
      MMA(0, 0); MMA(1, 0); MMA(2, 0); MMA(3, 0);
      ILV_PATTERN();
      __builtin_amdgcn_sched_barrier(0);
      WAIT_K(1, 0);
      lds_wr(CW(C_DONE + wave), (unsigned)(ci + 1));
      sb = (sb == (NS - 1) * STAGE) ? 0u : sb + STAGE;
      if (ci + 1 < NSTEP) {
        while (minf <= ci + 1) minf = lds_rd_min4(CW(C_FULL));
      }
      __builtin_amdgcn_sched_barrier(0);
      RDB(0, sb);
      MMA(0, 1); MMA(1, 1); MMA(2, 1); MMA(3, 1);
      ILV_PATTERN();
      __builtin_amdgcn_sched_barrier(0);
    }
  }

  if (wave == 0) STAMP(2);
  const unsigned seq = (unsigned)__builtin_amdgcn_readfirstlane(lds_rd(CW(C_SEQ)));
  bool failed = lds_rd(CW(C_FAIL)) != 0;
  if (wm != hk) {
    if (wn == 0) STAMP(6);
    char* pb = myws + WS_PART + ((size_t)(wg * 2 + wn) * 16) * 1024 + lane * 16;
#pragma unroll
    for (int i = 0; i < 4; ++i)
#pragma unroll
      for (int j = 0; j < 4; ++j) {
        char* a_ = pb + (i * 4 + j) * 1024;
        asm volatile("global_store_dwordx4 %0, %1, off sc1\n\ts_nop 1" :: "v"(a_), "v"(acc[i][j]) : "memory");
      }
    WAITV(0);
    if (lane == 0)
      __hip_atomic_store((unsigned*)(myws + WS_XFLAG + (size_t)(wg * 2 + wn) * 128), seq, __ATOMIC_RELAXED, __HIP_MEMORY_SCOPE_AGENT);
    if (wn == 0) STAMP(7);
    return;
  }
  {
    const int pwg = wg ^ 8;
    const unsigned* xf = (const unsigned*)(myws + WS_XFLAG + (size_t)(pwg * 2 + wn) * 128);
    const unsigned long long t0 = __builtin_amdgcn_s_memrealtime();
    unsigned spins = 0;
    while (!failed && __hip_atomic_load(xf, __ATOMIC_RELAXED, __HIP_MEMORY_SCOPE_AGENT) < seq) {
      if ((++spins & 63u) != 0u) continue;
      if (__builtin_amdgcn_s_memrealtime() - t0 > DEADLINE_TICKS ||
          __hip_atomic_load((unsigned*)(myws + WS_TMO), __ATOMIC_RELAXED, __HIP_MEMORY_SCOPE_AGENT) == seq) {
        __hip_atomic_store((unsigned*)(myws + WS_TMO), seq, __ATOMIC_RELAXED, __HIP_MEMORY_SCOPE_AGENT);
        failed = true;
      }
    }
    if (wn == 0) STAMP(15);
    const char* pb = myws + WS_PART + ((size_t)(pwg * 2 + wn) * 16) * 1024 + lane * 16;
    f32x4 t[16];
#pragma unroll
    for (int i = 0; i < 16; ++i) {
      const char* a_ = pb + i * 1024;
      asm volatile("global_load_dwordx4 %0, %1, off sc1" : "=&v"(t[i]) : "v"(a_) : "memory");
    }
    asm volatile("s_waitcnt vmcnt(0)"
                 : "+v"(t[0]), "+v"(t[1]), "+v"(t[2]), "+v"(t[3]), "+v"(t[4]), "+v"(t[5]), "+v"(t[6]), "+v"(t[7]),
                   "+v"(t[8]), "+v"(t[9]), "+v"(t[10]), "+v"(t[11]), "+v"(t[12]), "+v"(t[13]), "+v"(t[14]), "+v"(t[15]) :: "memory");
#pragma unroll
    for (int i = 0; i < 4; ++i)
#pragma unroll
      for (int j = 0; j < 4; ++j) acc[i][j] += t[i * 4 + j];
  }
  if (wn == 0) STAMP(1);

  float m = 0.f;
#pragma unroll
  for (int i = 0; i < 4; ++i)
#pragma unroll
    for (int j = 0; j < 4; ++j)
#pragma unroll
      for (int r = 0; r < 4; ++r) m = fmaxf(m, fabsf(acc[i][j][r]));
  {
    const unsigned mw = wave_max_u32(__float_as_uint(m));
    if (lane < NR)
      __hip_atomic_store(gdst, ((u64)seq << 32) | mw, __ATOMIC_RELAXED, __HIP_MEMORY_SCOPE_SYSTEM);
  }
  if (wn == 0) STAMP(3);
  unsigned gm = 0;
  {
    const u64* fp = (const u64*)(myws + WS_FIN + (wg & 7) * 128) + (lane & 7);
    const unsigned long long t0 = __builtin_amdgcn_s_memrealtime();
    unsigned spins = 0;
    while (!failed) {
      const u64 x = __hip_atomic_load(fp, __ATOMIC_RELAXED, __HIP_MEMORY_SCOPE_AGENT);
      if (__all((unsigned)(x >> 32) >= seq)) { gm = wave_max_u32((unsigned)x); failed = gm == 0xFFFFFFFFu; break; }
      __builtin_amdgcn_s_sleep(2);
      if ((++spins & 63u) != 0u) continue;
      if (__builtin_amdgcn_s_memrealtime() - t0 > DEADLINE_TICKS ||
          __hip_atomic_load((unsigned*)(myws + WS_TMO), __ATOMIC_RELAXED, __HIP_MEMORY_SCOPE_AGENT) == seq) {
        __hip_atomic_store((unsigned*)(myws + WS_TMO), seq, __ATOMIC_RELAXED, __HIP_MEMORY_SCOPE_AGENT);
        failed = true;
      }
    }
  }
  const float amax = __uint_as_float(gm);
  if (wn == 0) STAMP(4);
  float scale = amax / 127.0f;
  if (!(scale > 0.f)) scale = 1.0f;
  const float rscale = 1.0f / scale;

  signed char* tile = (signed char*)smem + wn * 4096;
#pragma unroll
  for (int mt = 0; mt < 4; ++mt)
#pragma unroll
    for (int nt = 0; nt < 4; ++nt) {
      unsigned pk = 0;
#pragma unroll
      for (int r = 0; r < 4; ++r) {
        const float y = acc[mt][nt][r];
        float q0 = y * rscale;
        q0 = fmaf(fmaf(-q0, scale, y), rscale, q0);
        float qv = rintf(q0);
        qv = fminf(fmaxf(qv, -127.f), 127.f);
        int qi = failed ? -128 : (int)qv;
        pk |= ((unsigned)qi & 0xFFu) << (8 * r);
      }
      *(unsigned*)(tile + (mt * 16 + fr) * 64 + nt * 16 + fq * 4) = pk;
    }
  WAITL0();
  __builtin_amdgcn_wave_barrier();
#pragma unroll
  for (int j = 0; j < 4; ++j) {
    const int c = lane + j * 64;
    const int row = c >> 2, cc = c & 3;
    u32x4 v = *(const u32x4*)((const char*)tile + row * 64 + cc * 16);
    *(u32x4*)(P.out + (size_t)(tm * BM + hk * 64 + row) * NDIM + tn * BN + wn * 64 + cc * 16) = v;
  }
  if (wn == 0) STAMP(5);
}

}

extern "C" void kernel_launch(int rank, int nranks, ncclComm_t comm,
                              hipStream_t stream,
                              void* const* d_in, const int* in_sizes, int n_in,
                              void* d_out, int out_size,
                              const int* in_dtypes, int out_dtype,
                              void* const* d_out_all,
                              void* const* d_ws_all, size_t ws_bytes) {
  (void)comm; (void)in_sizes; (void)n_in; (void)out_size; (void)in_dtypes; (void)out_dtype; (void)d_out_all;
  if (nranks != NR || ws_bytes < WS_NEED || (unsigned)rank >= (unsigned)NR) {
    fprintf(stderr, "unexpected configuration\n");
    return;
  }
  Params P;
  P.x = (const unsigned short*)d_in[0];
  P.w = (const unsigned short*)d_in[1];
  P.out = (signed char*)d_out;
  for (int p = 0; p < NR; ++p) P.ws[p] = (char*)d_ws_all[p];
  P.rank = rank;
  hipLaunchKernelGGL(a2a_gemm_q8, dim3(NWG), dim3(NTHREADS), 0, stream, P);
  HIP_CHECK(hipGetLastError());
}
```

```cpp
#include <hip/hip_runtime.h>
#include <rccl/rccl.h>
#include <cstdio>
#include <cstdint>

#define HIP_CHECK(expr) do { hipError_t _e = (expr); if (_e != hipSuccess) { \
    fprintf(stderr, "HIP error %s at %s:%d\n", hipGetErrorString(_e), __FILE__, __LINE__); \
    return; } } while (0)

typedef __attribute__((ext_vector_type(8))) short bf16x8;
typedef __attribute__((ext_vector_type(4))) short s16x4;
typedef __attribute__((ext_vector_type(4))) float f32x4;
typedef __attribute__((ext_vector_type(4))) unsigned u32x4;
typedef unsigned long long u64;

#define LAS __attribute__((address_space(3)))

namespace {

constexpr int NR = 8;
constexpr int MLOC = 512;
constexpr int KSH = 512;
constexpr int NDIM = 4096;
constexpr int BM = 128, BN = 128, BK = 64;
constexpr int NS = 4;
constexpr int A_BYTES = BM * BK * 2;
constexpr int B_BYTES = BK * BN * 2;
constexpr int STAGE = A_BYTES + B_BYTES;
constexpr int NWG = 256;
constexpr int NSTEP = 32;
constexpr int NOWN = 4;
constexpr int NREM = NSTEP - NOWN;
constexpr int PUSHW = 32;
constexpr int NTHREADS = 640;
#define PACE_N 4
#define PACE_FROM 2
#define PUSH_T 170

constexpr size_t WS_A = 0;
constexpr size_t WS_FLAG = 4u << 20;
constexpr size_t WS_AMAX = (4u << 20) + (64u << 10);
constexpr size_t WS_CNT = (4u << 20) + (128u << 10);
constexpr size_t WS_TMO = WS_CNT + 4096;
constexpr size_t WS_FIN = (4u << 20) + (160u << 10);
constexpr size_t WS_XFLAG = (4u << 20) + (192u << 10);
constexpr size_t WS_PART = (6u << 20);
constexpr size_t WS_NEED = WS_PART + (8u << 20);
#define STAMP(i) do { } while (0)

constexpr unsigned long long DEADLINE_TICKS = 40000000ull;

constexpr int C_FULL = 0;
constexpr int C_DONE = 4;
constexpr int C_RDY = 8;
constexpr int C_SEQ = 9;
constexpr int C_FAIL = 10;
constexpr int C_QUEUE = 32;

struct Params {
  const unsigned short* x;
  const unsigned short* w;
  signed char* out;
  char* ws[NR];
  int rank;
};

__device__ __forceinline__ void bar() { __builtin_amdgcn_s_barrier(); }
#define WAITV(n) asm volatile("s_waitcnt vmcnt(" #n ")" ::: "memory")
#define WAITL0() asm volatile("s_waitcnt lgkmcnt(0)" ::: "memory")

__device__ __forceinline__ void lds_wr(unsigned addr, unsigned v) {
  asm volatile("ds_write_b32 %0, %1" :: "v"(addr), "v"(v) : "memory");
}
__device__ __forceinline__ unsigned lds_rd(unsigned addr) {
  unsigned v;
  asm volatile("ds_read_b32 %0, %1\n\ts_waitcnt lgkmcnt(0)" : "=&v"(v) : "v"(addr) : "memory");
  return v;
}
__device__ __forceinline__ int lds_rd_min4(unsigned addr) {
  u32x4 v;
  asm volatile("ds_read_b128 %0, %1\n\ts_waitcnt lgkmcnt(0)" : "=&v"(v) : "v"(addr) : "memory");
  unsigned m = min(min(v[0], v[1]), min(v[2], v[3]));
  return __builtin_amdgcn_readfirstlane((int)m);
}
__device__ __forceinline__ unsigned wave_max_u32(unsigned v) {
  v = max(v, (unsigned)__builtin_amdgcn_update_dpp(0, (int)v, 0x111, 0xF, 0xF, false));
  v = max(v, (unsigned)__builtin_amdgcn_update_dpp(0, (int)v, 0x112, 0xF, 0xF, false));
  v = max(v, (unsigned)__builtin_amdgcn_update_dpp(0, (int)v, 0x114, 0xF, 0xF, false));
  v = max(v, (unsigned)__builtin_amdgcn_update_dpp(0, (int)v, 0x118, 0xF, 0xF, false));
  v = max(v, (unsigned)__builtin_amdgcn_update_dpp(0, (int)v, 0x142, 0xA, 0xF, false));
  v = max(v, (unsigned)__builtin_amdgcn_update_dpp(0, (int)v, 0x143, 0xC, 0xF, false));
  return (unsigned)__builtin_amdgcn_readlane((int)v, 63);
}
#define VM0_KNOWN() __builtin_amdgcn_s_waitcnt(0x0F70)

__device__ __forceinline__ void glds16(const void* g, char* l) {
  __builtin_amdgcn_global_load_lds((const __attribute__((address_space(1))) void*)g, (LAS void*)l, 16, 0, 0);
}

__global__ __launch_bounds__(NTHREADS, 3) void a2a_gemm_q8(Params P) {
  __shared__ __attribute__((aligned(16))) char smem[NS * STAGE + 1024 + 16384];
  const int tid = threadIdx.x;
  const int wave = __builtin_amdgcn_readfirstlane(tid >> 6);
  const int lane = tid & 63;
  const int wg = blockIdx.x;
  const int rank = P.rank;
  const unsigned lds0 = (unsigned)(size_t)(LAS char*)smem;
  const unsigned ctrl = lds0 + NS * STAGE;
#define CW(i) (ctrl + 4u * (unsigned)(i))
  char* myws = P.ws[rank];
  const int xg = wg & 7, idx = wg >> 3;
  const int hk = idx & 1;
  const int tm = (idx >> 1) & 3, tn = xg * 4 + (idx >> 3);

  if (wave == 8) {
    const unsigned long long tstart = __builtin_amdgcn_s_memrealtime();
    const bool active = wg < (NR - 1) * PUSHW;
    const int dst = (rank + 1 + wg / PUSHW) & (NR - 1);
    const int pw = wg % PUSHW;
    const int ps = pw >> 2, ptm = pw & 3;
    u32x4 d[16];
    const int r8 = lane >> 3, ch = lane & 7;
    if (active) {
      const unsigned short* src = P.x + (size_t)(dst * MLOC + ptm * 128 + r8) * KSH + ps * 64 + ch * 8;
#pragma unroll
      for (int j = 0; j < 16; ++j) d[j] = *(const u32x4*)(src + (size_t)j * 8 * KSH);
    }
    bar();
    VM0_KNOWN();
    int pf = 0;
    auto prefetch_tile = [&](int i4) {
      const int i = i4 * 4 + tm;
      const int sp = i / 7, j = i - sp * 7;
      const int p = (rank + 1 + j) & 7, s2 = sp * 2 + (hk ^ (p & 1));
      const char* bb = (const char*)P.w + (size_t)(p * KSH + s2 * 64 + (lane >> 4)) * (NDIM * 2) + tn * (BN * 2) + (lane & 15) * 16;
      char* scr = smem + NS * STAGE + 1024;
#pragma unroll
      for (int q = 0; q < 16; ++q) glds16(bb + (size_t)q * 4 * (NDIM * 2), scr + q * 1024 + lane * 16);
      WAITV(16);
    };
    if (!active) { for (; pf < 7; ++pf) prefetch_tile(pf); WAITV(0); return; }
    while (pf < 7 && __builtin_amdgcn_s_memrealtime() - tstart + 150 < (unsigned long long)(ps * PUSH_T)) { prefetch_tile(pf); ++pf; }
    WAITV(0);
    while (__builtin_amdgcn_s_memrealtime() - tstart < (unsigned long long)(ps * PUSH_T)) __builtin_amdgcn_s_sleep(2);
    char* dbase = P.ws[dst] + WS_A + ((size_t)(rank * 8 + ps) * MLOC + ptm * 128 + r8) * 128 + ch * 16;
#pragma unroll
    for (int j = 0; j < 16; ++j) {
      char* a_ = dbase + j * 1024;
      asm volatile("global_store_dwordx4 %0, %1, off sc0 sc1\n\ts_nop 1" :: "v"(a_), "v"(d[j]) : "memory");
      if ((j % PACE_N) == PACE_N - 1 && ps >= PACE_FROM) WAITV(0);
    }
    WAITV(0);
    unsigned seq;
    do { seq = (unsigned)__builtin_amdgcn_readfirstlane(lds_rd(CW(C_SEQ))); } while (seq == 0);
    unsigned* fl = (unsigned*)(P.ws[dst] + WS_FLAG) + ptm * 64 + rank * 8 + ps;
    if (lane == 0) asm volatile("global_store_dword %0, %1, off sc0 sc1" :: "v"(fl), "v"(seq) : "memory");
    for (; pf < 7; ++pf) prefetch_tile(pf);
    WAITV(0);
    return;
  }

  if (wave == 9) {
    if (lane < 12) lds_wr(CW(lane), 0);
    WAITL0();
    bar();
    unsigned ticket = 0;
    if (lane == 0)
      ticket = __hip_atomic_fetch_add((unsigned*)(myws + WS_CNT + (wg & 15) * 128), 1u, __ATOMIC_RELAXED, __HIP_MEMORY_SCOPE_AGENT);
    const unsigned seq = (unsigned)__builtin_amdgcn_readfirstlane(ticket) / (NWG / 16) + 1u;
    lds_wr(CW(C_SEQ), seq);
    const unsigned* fp = (const unsigned*)(myws + WS_FLAG) + tm * 64 + lane;
    const unsigned long long t0 = __builtin_amdgcn_s_memrealtime();
    unsigned long long seen = (0xFFull << (rank * 8)) | (hk ? 0xAA55AA55AA55AA55ull : 0x55AA55AA55AA55AAull);
    int cnt = 0;
    unsigned spins = 0;
    while (cnt < NREM) {
      const unsigned mn = __hip_atomic_load(fp, __ATOMIC_RELAXED, __HIP_MEMORY_SCOPE_SYSTEM);
      const unsigned long long rdy = __ballot(mn >= seq);
      const unsigned long long nw = rdy & ~seen;
      if (nw) {
        if ((nw >> lane) & 1ull) {
          const int pos = cnt + __popcll(nw & ((1ull << lane) - 1ull));
          lds_wr(CW(C_QUEUE + pos), (unsigned)lane);
        }
        WAITL0();
        cnt += __popcll(nw);
        seen |= nw;
        lds_wr(CW(C_RDY), (unsigned)cnt);
      } else if ((++spins & 63u) == 0u) {
        if (__builtin_amdgcn_s_memrealtime() - t0 > DEADLINE_TICKS ||
            __hip_atomic_load((unsigned*)(myws + WS_TMO), __ATOMIC_RELAXED, __HIP_MEMORY_SCOPE_AGENT) == seq) {
          if (lane == 0)
            __hip_atomic_store((unsigned*)(myws + WS_TMO), seq, __ATOMIC_RELAXED, __HIP_MEMORY_SCOPE_AGENT);
          lds_wr(CW(C_FAIL), 1);
          if (lane < NREM) lds_wr(CW(C_QUEUE + lane), (unsigned)(rank * 8));
          WAITL0();
          lds_wr(CW(C_RDY), NREM);
          cnt = NREM;
        }
      }
    }
    if (wg < 8) {
      const u64* gp = (const u64*)(myws + WS_AMAX) + wg * 512 + lane;
      bool ok = false;
      unsigned gl = 0, pass = 0;
      for (;; ++pass) {
        if (!ok) {
          ok = true; gl = 0;
#pragma unroll
          for (int k = 0; k < 8; ++k) {
            u64 x = __hip_atomic_load(gp + k * 64, __ATOMIC_RELAXED, __HIP_MEMORY_SCOPE_SYSTEM);
            ok &= (unsigned)(x >> 32) >= seq;
            gl = max(gl, (unsigned)x);
          }
        }
        if (__all(ok)) break;
        if ((pass & 31u) == 31u &&
            (__builtin_amdgcn_s_memrealtime() - t0 > DEADLINE_TICKS ||
             __hip_atomic_load((unsigned*)(myws + WS_TMO), __ATOMIC_RELAXED, __HIP_MEMORY_SCOPE_AGENT) == seq)) {
          if (lane == 0)
            __hip_atomic_store((unsigned*)(myws + WS_TMO), seq, __ATOMIC_RELAXED, __HIP_MEMORY_SCOPE_AGENT);
          gl = 0xFFFFFFFFu;
          break;
        }
      }
      gl = wave_max_u32(gl);
      if (lane < 8)
        __hip_atomic_store((u64*)(myws + WS_FIN + lane * 128) + wg, ((u64)seq << 32) | gl, __ATOMIC_RELAXED, __HIP_MEMORY_SCOPE_AGENT);
    }
    return;
  }

  if (wave >= 4) {
    const int lw = wave - 4;
    const int r8 = lane >> 3;
    const int chA = (lane & 7) ^ r8;
    const unsigned offA_own = (unsigned)(tm * BM + lw * 32 + r8) * 1024u + chA * 16;
    const unsigned offA_stg = (unsigned)(tm * BM + lw * 32 + r8) * 128u + chA * 16;
    unsigned offB[4];
#pragma unroll
    for (int j = 0; j < 4; ++j) {
      const int k = (lw * 4 + j) * 4 + (lane >> 4);
      const int f = (k & 3) | (((k >> 3) & 1) << 2);
      const int sl = lane & 15;
      const int ch = (((sl >> 1) ^ f) << 1) | (sl & 1);
      offB[j] = (unsigned)k * (NDIM * 2) + tn * (BN * 2) + ch * 16;
    }
    const char* xown = (const char*)P.x + (size_t)rank * MLOC * 1024;
    const char* wsA = myws + WS_A;
    const char* wbase = (const char*)P.w;
    int li = 0, pub = 0, slot = 0, avail = NOWN, mind = 0, unext = 0;
    while (li < NSTEP) {
      while (li >= avail || li >= mind + NS) {
        if (li - pub == 2) { WAITV(8); ++pub; lds_wr(CW(C_FULL + lw), (unsigned)pub); }
        else if (li - pub == 1) { WAITV(0); ++pub; lds_wr(CW(C_FULL + lw), (unsigned)pub); }
        mind = lds_rd_min4(CW(C_DONE));
        avail = NOWN + __builtin_amdgcn_readfirstlane((int)lds_rd(CW(C_RDY)));
        unext = __builtin_amdgcn_readfirstlane((int)lds_rd(CW(C_QUEUE + (li >= NOWN ? li - NOWN : 0))));
      }
      int p, s;
      const char* abase;
      unsigned aoffs, astep;
      if (li < NOWN) {
        p = rank; s = 2 * li + (hk ^ (rank & 1));
        abase = xown + s * 128; aoffs = offA_own; astep = 8 * 1024;
      } else {
        p = unext >> 3; s = unext & 7;
        abase = wsA + (size_t)unext * 65536; aoffs = offA_stg; astep = 1024;
      }
      char* sb = smem + slot * STAGE;
#pragma unroll
      for (int j = 0; j < 4; ++j)
        glds16(abase + aoffs + j * astep, sb + (lw * 4 + j) * 1024 + lane * 16);
      const char* bb = wbase + (size_t)(p * KSH + s * 64) * (NDIM * 2);
#pragma unroll
      for (int j = 0; j < 4; ++j)
        glds16(bb + offB[j], sb + A_BYTES + (lw * 4 + j) * 1024 + lane * 16);
      ++li;
      slot = (slot == NS - 1) ? 0 : slot + 1;
      if (li == 1) bar();
      u32x4 vd; unsigned vr, vq;
      asm volatile("ds_read_b128 %0, %3\n\tds_read_b32 %1, %4\n\tds_read_b32 %2, %5"
                   : "=&v"(vd), "=&v"(vr), "=&v"(vq)
                   : "v"(CW(C_DONE)), "v"(CW(C_RDY)), "v"(CW(C_QUEUE + (li >= NOWN ? li - NOWN : 0))) : "memory");
      if (li - pub == 3) { WAITV(16); ++pub; lds_wr(CW(C_FULL + lw), (unsigned)pub); }
      asm volatile("s_waitcnt lgkmcnt(0)" : "+v"(vd), "+v"(vr), "+v"(vq) :: "memory");
      mind = __builtin_amdgcn_readfirstlane((int)min(min(vd[0], vd[1]), min(vd[2], vd[3])));
      avail = NOWN + __builtin_amdgcn_readfirstlane((int)vr);
      unext = __builtin_amdgcn_readfirstlane((int)vq);
    }
    WAITV(8); lds_wr(CW(C_FULL + lw), (unsigned)(NSTEP - 1));
    WAITV(0); lds_wr(CW(C_FULL + lw), (unsigned)NSTEP);
    return;
  }

  const int wm = wave >> 1, wn = wave & 1;
  const int fr = lane & 15, fq = lane >> 4;
  unsigned aoff[2];
#pragma unroll
  for (int kk = 0; kk < 2; ++kk)
    aoff[kk] = lds0 + (unsigned)(wm * 64 + fr) * 128u + (((kk * 4 + fq) ^ (lane & 7)) * 16);
  unsigned boff[4];
  {
    const int q = (lane & 15) >> 2, p = lane & 3, g = fq;
    const int f = q | ((g & 1) << 2);
#pragma unroll
    for (int nt = 0; nt < 4; ++nt)
      boff[nt] = lds0 + A_BYTES + (unsigned)(8 * g + q) * 256u + (((wn * 4 + nt) ^ f) * 32) + p * 8;
  }

  u64* gdst = (u64*)(P.ws[lane & (NR - 1)] + WS_AMAX) + (rank * NWG + wg) * 2 + wn;
  f32x4 acc[4][4];
#pragma unroll
  for (int i = 0; i < 4; ++i)
#pragma unroll
    for (int j = 0; j < 4; ++j) acc[i][j] = f32x4{0.f, 0.f, 0.f, 0.f};

  bar();
  if (wave == 0) STAMP(0);
  {
    bf16x8 a[4][2];
    s16x4 bl[4][2], bh[4][2];
#define WAIT_K(kk, n) asm volatile("s_waitcnt lgkmcnt(" #n ")" \
                   : "+v"(bl[0][kk]), "+v"(bh[0][kk]), "+v"(bl[1][kk]), "+v"(bh[1][kk]), \
                     "+v"(bl[2][kk]), "+v"(bh[2][kk]), "+v"(bl[3][kk]), "+v"(bh[3][kk]), \
                     "+v"(a[0][kk]), "+v"(a[1][kk]), "+v"(a[2][kk]), "+v"(a[3][kk]) :: "memory")
#define BFRAG(nt, kk) bf16x8{bl[nt][kk][0], bl[nt][kk][1], bl[nt][kk][2], bl[nt][kk][3], \
                             bh[nt][kk][0], bh[nt][kk][1], bh[nt][kk][2], bh[nt][kk][3]}
#define MMA(nt, kk) do { const bf16x8 bb_ = BFRAG(nt, kk); _Pragma("unroll") for (int mt = 0; mt < 4; ++mt) \
        acc[mt][nt] = __builtin_amdgcn_mfma_f32_16x16x32_bf16(bb_, a[mt][kk], acc[mt][nt], 0, 0, 0); } while (0)
    typedef short v4i16_t __attribute__((ext_vector_type(4)));
#define LDT(addr) __builtin_bit_cast(s16x4, __builtin_amdgcn_ds_read_tr16_b64_v4i16((LAS v4i16_t*)(size_t)(addr)))
#define LDA(addr) (*(const LAS bf16x8*)(size_t)(addr))
#define RDB(kk, sb_) do { _Pragma("unroll") for (int nt = 0; nt < 4; ++nt) { \
        bl[nt][kk] = LDT((sb_) + boff[nt] + (kk) * 8192); bh[nt][kk] = LDT((sb_) + boff[nt] + (kk) * 8192 + 1024); } \
        _Pragma("unroll") for (int mt = 0; mt < 4; ++mt) a[mt][kk] = LDA((sb_) + aoff[kk] + mt * 2048); } while (0)
#define ILV_PATTERN() do { _Pragma("unroll") for (int q = 0; q < 12; ++q) { \
        __builtin_amdgcn_sched_group_barrier(0x8, 1, 0); __builtin_amdgcn_sched_group_barrier(0x100, 1, 0); } \
        __builtin_amdgcn_sched_group_barrier(0x8, 4, 0); } while (0)
    int minf = 0;
    unsigned sb = 0;
    while (minf <= 0) minf = lds_rd_min4(CW(C_FULL));
    RDB(0, sb);
    __builtin_amdgcn_sched_barrier(0);
    for (int ci = 0; ci < NSTEP; ++ci) {
      RDB(1, sb);
      MMA(0, 0); MMA(1, 0); MMA(2, 0); MMA(3, 0);
      ILV_PATTERN();
      __builtin_amdgcn_sched_barrier(0);
      WAIT_K(1, 0);
      lds_wr(CW(C_DONE + wave), (unsigned)(ci + 1));
      sb = (sb == (NS - 1) * STAGE) ? 0u : sb + STAGE;
      if (ci + 1 < NSTEP) {
        while (minf <= ci + 1) minf = lds_rd_min4(CW(C_FULL));
      }
      __builtin_amdgcn_sched_barrier(0);
      RDB(0, sb);
      MMA(0, 1); MMA(1, 1); MMA(2, 1); MMA(3, 1);
      ILV_PATTERN();
      __builtin_amdgcn_sched_barrier(0);
    }
  }

  if (wave == 0) STAMP(2);
  const unsigned seq = (unsigned)__builtin_amdgcn_readfirstlane(lds_rd(CW(C_SEQ)));
  bool failed = lds_rd(CW(C_FAIL)) != 0;
  if (wm != hk) {
    if (wn == 0) STAMP(6);
    char* pb = myws + WS_PART + ((size_t)(wg * 2 + wn) * 16) * 1024 + lane * 16;
#pragma unroll
    for (int i = 0; i < 4; ++i)
#pragma unroll
      for (int j = 0; j < 4; ++j) {
        char* a_ = pb + (i * 4 + j) * 1024;
        asm volatile("global_store_dwordx4 %0, %1, off sc1\n\ts_nop 1" :: "v"(a_), "v"(acc[i][j]) : "memory");
      }
    WAITV(0);
    if (lane == 0)
      __hip_atomic_store((unsigned*)(myws + WS_XFLAG + (size_t)(wg * 2 + wn) * 128), seq, __ATOMIC_RELAXED, __HIP_MEMORY_SCOPE_AGENT);
    if (wn == 0) STAMP(7);
    return;
  }
  {
    const int pwg = wg ^ 8;
    const unsigned* xf = (const unsigned*)(myws + WS_XFLAG + (size_t)(pwg * 2 + wn) * 128);
    const unsigned long long t0 = __builtin_amdgcn_s_memrealtime();
    unsigned spins = 0;
    while (!failed && __hip_atomic_load(xf, __ATOMIC_RELAXED, __HIP_MEMORY_SCOPE_AGENT) < seq) {
      if ((++spins & 63u) != 0u) continue;
      if (__builtin_amdgcn_s_memrealtime() - t0 > DEADLINE_TICKS ||
          __hip_atomic_load((unsigned*)(myws + WS_TMO), __ATOMIC_RELAXED, __HIP_MEMORY_SCOPE_AGENT) == seq) {
        __hip_atomic_store((unsigned*)(myws + WS_TMO), seq, __ATOMIC_RELAXED, __HIP_MEMORY_SCOPE_AGENT);
        failed = true;
      }
    }
    if (wn == 0) STAMP(15);
    const char* pb = myws + WS_PART + ((size_t)(pwg * 2 + wn) * 16) * 1024 + lane * 16;
    f32x4 t[16];
#pragma unroll
    for (int i = 0; i < 16; ++i) {
      const char* a_ = pb + i * 1024;
      asm volatile("global_load_dwordx4 %0, %1, off sc1" : "=&v"(t[i]) : "v"(a_) : "memory");
    }
    asm volatile("s_waitcnt vmcnt(0)"
                 : "+v"(t[0]), "+v"(t[1]), "+v"(t[2]), "+v"(t[3]), "+v"(t[4]), "+v"(t[5]), "+v"(t[6]), "+v"(t[7]),
                   "+v"(t[8]), "+v"(t[9]), "+v"(t[10]), "+v"(t[11]), "+v"(t[12]), "+v"(t[13]), "+v"(t[14]), "+v"(t[15]) :: "memory");
#pragma unroll
    for (int i = 0; i < 4; ++i)
#pragma unroll
      for (int j = 0; j < 4; ++j) acc[i][j] += t[i * 4 + j];
  }
  if (wn == 0) STAMP(1);

  float m = 0.f;
#pragma unroll
  for (int i = 0; i < 4; ++i)
#pragma unroll
    for (int j = 0; j < 4; ++j)
#pragma unroll
      for (int r = 0; r < 4; ++r) m = fmaxf(m, fabsf(acc[i][j][r]));
  {
    const unsigned mw = wave_max_u32(__float_as_uint(m));
    if (lane < NR)
      __hip_atomic_store(gdst, ((u64)seq << 32) | mw, __ATOMIC_RELAXED, __HIP_MEMORY_SCOPE_SYSTEM);
  }
  if (wn == 0) STAMP(3);
  unsigned gm = 0;
  {
    const u64* fp = (const u64*)(myws + WS_FIN + (wg & 7) * 128) + (lane & 7);
    const unsigned long long t0 = __builtin_amdgcn_s_memrealtime();
    unsigned spins = 0;
    while (!failed) {
      const u64 x = __hip_atomic_load(fp, __ATOMIC_RELAXED, __HIP_MEMORY_SCOPE_AGENT);
      if (__all((unsigned)(x >> 32) >= seq)) { gm = wave_max_u32((unsigned)x); failed = gm == 0xFFFFFFFFu; break; }
      __builtin_amdgcn_s_sleep(2);
      if ((++spins & 63u) != 0u) continue;
      if (__builtin_amdgcn_s_memrealtime() - t0 > DEADLINE_TICKS ||
          __hip_atomic_load((unsigned*)(myws + WS_TMO), __ATOMIC_RELAXED, __HIP_MEMORY_SCOPE_AGENT) == seq) {
        __hip_atomic_store((unsigned*)(myws + WS_TMO), seq, __ATOMIC_RELAXED, __HIP_MEMORY_SCOPE_AGENT);
        failed = true;
      }
    }
  }
  const float amax = __uint_as_float(gm);
  if (wn == 0) STAMP(4);
  float scale = amax / 127.0f;
  if (!(scale > 0.f)) scale = 1.0f;
  const float rscale = 1.0f / scale;

  signed char* tile = (signed char*)smem + wn * 4096;
#pragma unroll
  for (int mt = 0; mt < 4; ++mt)
#pragma unroll
    for (int nt = 0; nt < 4; ++nt) {
      unsigned pk = 0;
#pragma unroll
      for (int r = 0; r < 4; ++r) {
        const float y = acc[mt][nt][r];
        float q0 = y * rscale;
        q0 = fmaf(fmaf(-q0, scale, y), rscale, q0);
        float qv = rintf(q0);
        qv = fminf(fmaxf(qv, -127.f), 127.f);
        int qi = failed ? -128 : (int)qv;
        pk |= ((unsigned)qi & 0xFFu) << (8 * r);
      }
      *(unsigned*)(tile + (mt * 16 + fr) * 64 + nt * 16 + fq * 4) = pk;
    }
  WAITL0();
  __builtin_amdgcn_wave_barrier();
#pragma unroll
  for (int j = 0; j < 4; ++j) {
    const int c = lane + j * 64;
    const int row = c >> 2, cc = c & 3;
    u32x4 v = *(const u32x4*)((const char*)tile + row * 64 + cc * 16);
    *(u32x4*)(P.out + (size_t)(tm * BM + hk * 64 + row) * NDIM + tn * BN + wn * 64 + cc * 16) = v;
  }
  if (wn == 0) STAMP(5);
}

}

extern "C" void kernel_launch(int rank, int nranks, ncclComm_t comm,
                              hipStream_t stream,
                              void* const* d_in, const int* in_sizes, int n_in,
                              void* d_out, int out_size,
                              const int* in_dtypes, int out_dtype,
                              void* const* d_out_all,
                              void* const* d_ws_all, size_t ws_bytes) {
  (void)comm; (void)in_sizes; (void)n_in; (void)out_size; (void)in_dtypes; (void)out_dtype; (void)d_out_all;
  if (nranks != NR || ws_bytes < WS_NEED || (unsigned)rank >= (unsigned)NR) {
    fprintf(stderr, "unexpected configuration\n");
    return;
  }
  Params P;
  P.x = (const unsigned short*)d_in[0];
  P.w = (const unsigned short*)d_in[1];
  P.out = (signed char*)d_out;
  for (int p = 0; p < NR; ++p) P.ws[p] = (char*)d_ws_all[p];
  P.rank = rank;
  hipLaunchKernelGGL(a2a_gemm_q8, dim3(NWG), dim3(NTHREADS), 0, stream, P);
  HIP_CHECK(hipGetLastError());
}
```

```cpp
#include <hip/hip_runtime.h>
#include <rccl/rccl.h>
#include <cstdio>
#include <cstdint>

#define HIP_CHECK(expr) do { hipError_t _e = (expr); if (_e != hipSuccess) { \
    fprintf(stderr, "HIP error %s at %s:%d\n", hipGetErrorString(_e), __FILE__, __LINE__); \
    return; } } while (0)

typedef __attribute__((ext_vector_type(8))) short bf16x8;
typedef __attribute__((ext_vector_type(4))) short s16x4;
typedef __attribute__((ext_vector_type(4))) float f32x4;
typedef __attribute__((ext_vector_type(4))) unsigned u32x4;
typedef unsigned long long u64;

#define LAS __attribute__((address_space(3)))

namespace {

constexpr int NR = 8;
constexpr int MLOC = 512;
constexpr int KSH = 512;
constexpr int NDIM = 4096;
constexpr int BM = 128, BN = 128, BK = 64;
constexpr int NS = 4;
constexpr int A_BYTES = BM * BK * 2;
constexpr int B_BYTES = BK * BN * 2;
constexpr int STAGE = A_BYTES + B_BYTES;
constexpr int NWG = 256;
constexpr int NSTEP = 32;
constexpr int NOWN = 4;
constexpr int NREM = NSTEP - NOWN;
constexpr int PUSHW = 32;
constexpr int NTHREADS = 640;
#define PACE_N 4
#define PACE_FROM 2
#define PUSH_T 170

constexpr size_t WS_A = 0;
constexpr size_t WS_FLAG = 4u << 20;
constexpr size_t WS_AMAX = (4u << 20) + (64u << 10);
constexpr size_t WS_CNT = (4u << 20) + (128u << 10);
constexpr size_t WS_TMO = WS_CNT + 4096;
constexpr size_t WS_FIN = (4u << 20) + (160u << 10);
constexpr size_t WS_XFLAG = (4u << 20) + (192u << 10);
constexpr size_t WS_PART = (6u << 20);
constexpr size_t WS_NEED = WS_PART + (8u << 20);
#define STAMP(i) do { } while (0)

constexpr unsigned long long DEADLINE_TICKS = 40000000ull;

constexpr int C_FULL = 0;
constexpr int C_DONE = 4;
constexpr int C_RDY = 8;
constexpr int C_SEQ = 9;
constexpr int C_FAIL = 10;
constexpr int C_QUEUE = 32;

struct Params {
  const unsigned short* x;
  const unsigned short* w;
  signed char* out;
  char* ws[NR];
  int rank;
};

__device__ __forceinline__ void bar() { __builtin_amdgcn_s_barrier(); }
#define WAITV(n) asm volatile("s_waitcnt vmcnt(" #n ")" ::: "memory")
#define WAITL0() asm volatile("s_waitcnt lgkmcnt(0)" ::: "memory")

__device__ __forceinline__ void lds_wr(unsigned addr, unsigned v) {
  asm volatile("ds_write_b32 %0, %1" :: "v"(addr), "v"(v) : "memory");
}
__device__ __forceinline__ unsigned lds_rd(unsigned addr) {
  unsigned v;
  asm volatile("ds_read_b32 %0, %1\n\ts_waitcnt lgkmcnt(0)" : "=&v"(v) : "v"(addr) : "memory");
  return v;
}
__device__ __forceinline__ int lds_rd_min4(unsigned addr) {
  u32x4 v;
  asm volatile("ds_read_b128 %0, %1\n\ts_waitcnt lgkmcnt(0)" : "=&v"(v) : "v"(addr) : "memory");
  unsigned m = min(min(v[0], v[1]), min(v[2], v[3]));
  return __builtin_amdgcn_readfirstlane((int)m);
}
__device__ __forceinline__ unsigned wave_max_u32(unsigned v) {
  v = max(v, (unsigned)__builtin_amdgcn_update_dpp(0, (int)v, 0x111, 0xF, 0xF, false));
  v = max(v, (unsigned)__builtin_amdgcn_update_dpp(0, (int)v, 0x112, 0xF, 0xF, false));
  v = max(v, (unsigned)__builtin_amdgcn_update_dpp(0, (int)v, 0x114, 0xF, 0xF, false));
  v = max(v, (unsigned)__builtin_amdgcn_update_dpp(0, (int)v, 0x118, 0xF, 0xF, false));
  v = max(v, (unsigned)__builtin_amdgcn_update_dpp(0, (int)v, 0x142, 0xA, 0xF, false));
  v = max(v, (unsigned)__builtin_amdgcn_update_dpp(0, (int)v, 0x143, 0xC, 0xF, false));
  return (unsigned)__builtin_amdgcn_readlane((int)v, 63);
}
#define VM0_KNOWN() __builtin_amdgcn_s_waitcnt(0x0F70)

__device__ __forceinline__ void glds16(const void* g, char* l) {
  __builtin_amdgcn_global_load_lds((const __attribute__((address_space(1))) void*)g, (LAS void*)l, 16, 0, 0);
}

__global__ __launch_bounds__(NTHREADS, 3) void a2a_gemm_q8(Params P) {
  __shared__ __attribute__((aligned(16))) char smem[NS * STAGE + 512];
  const int tid = threadIdx.x;
  const int wave = __builtin_amdgcn_readfirstlane(tid >> 6);
  const int lane = tid & 63;
  const int wg = blockIdx.x;
  const int rank = P.rank;
  const unsigned lds0 = (unsigned)(size_t)(LAS char*)smem;
  const unsigned ctrl = lds0 + NS * STAGE;
#define CW(i) (ctrl + 4u * (unsigned)(i))
  char* myws = P.ws[rank];
  const int xg = wg & 7, idx = wg >> 3;
  const int hk = idx & 1;
  const int tm = (idx >> 1) & 3, tn = xg * 4 + (idx >> 3);

  if (wave == 8) {
    const unsigned long long tstart = __builtin_amdgcn_s_memrealtime();
    const bool active = wg < (NR - 1) * PUSHW;
    const int dst = (rank + 1 + wg / PUSHW) & (NR - 1);
    const int pw = wg % PUSHW;
    const int ps = pw >> 2, ptm = pw & 3;
    u32x4 d[16];
    const int r8 = lane >> 3, ch = lane & 7;
    if (active) {
      const unsigned short* src = P.x + (size_t)(dst * MLOC + ptm * 128 + r8) * KSH + ps * 64 + ch * 8;
#pragma unroll
      for (int j = 0; j < 16; ++j) d[j] = *(const u32x4*)(src + (size_t)j * 8 * KSH);
    }
    bar();
    if (!active) return;
    VM0_KNOWN();
    while (__builtin_amdgcn_s_memrealtime() - tstart < (unsigned long long)(ps * PUSH_T)) __builtin_amdgcn_s_sleep(2);
    char* dbase = P.ws[dst] + WS_A + ((size_t)(rank * 8 + ps) * MLOC + ptm * 128 + r8) * 128 + ch * 16;
#pragma unroll
    for (int j = 0; j < 16; ++j) {
      char* a_ = dbase + j * 1024;
      asm volatile("global_store_dwordx4 %0, %1, off sc0 sc1\n\ts_nop 1" :: "v"(a_), "v"(d[j]) : "memory");
      if ((j % PACE_N) == PACE_N - 1 && ps >= PACE_FROM) WAITV(0);
    }
    WAITV(0);
    unsigned seq;
    do { seq = (unsigned)__builtin_amdgcn_readfirstlane(lds_rd(CW(C_SEQ))); } while (seq == 0);
    unsigned* fl = (unsigned*)(P.ws[dst] + WS_FLAG) + ptm * 64 + rank * 8 + ps;
    if (lane == 0) asm volatile("global_store_dword %0, %1, off sc0 sc1" :: "v"(fl), "v"(seq) : "memory");
    return;
  }

  if (wave == 9) {
    if (lane < 12) lds_wr(CW(lane), 0);
    WAITL0();
    bar();
    unsigned ticket = 0;
    if (lane == 0)
      ticket = __hip_atomic_fetch_add((unsigned*)(myws + WS_CNT + (wg & 15) * 128), 1u, __ATOMIC_RELAXED, __HIP_MEMORY_SCOPE_AGENT);
    const unsigned seq = (unsigned)__builtin_amdgcn_readfirstlane(ticket) / (NWG / 16) + 1u;
    lds_wr(CW(C_SEQ), seq);
    const unsigned* fp = (const unsigned*)(myws + WS_FLAG) + tm * 64 + lane;
    const unsigned long long t0 = __builtin_amdgcn_s_memrealtime();
    unsigned long long seen = (0xFFull << (rank * 8)) | (hk ? 0xAA55AA55AA55AA55ull : 0x55AA55AA55AA55AAull);
    int cnt = 0;
    unsigned spins = 0;
    while (cnt < NREM) {
      const unsigned mn = __hip_atomic_load(fp, __ATOMIC_RELAXED, __HIP_MEMORY_SCOPE_SYSTEM);
      const unsigned long long rdy = __ballot(mn >= seq);
      const unsigned long long nw = rdy & ~seen;
      if (nw) {
        if ((nw >> lane) & 1ull) {
          const int pos = cnt + __popcll(nw & ((1ull << lane) - 1ull));
          lds_wr(CW(C_QUEUE + pos), (unsigned)lane);
        }
        WAITL0();
        cnt += __popcll(nw);
        seen |= nw;
        lds_wr(CW(C_RDY), (unsigned)cnt);
      } else if ((++spins & 63u) == 0u) {
        if (__builtin_amdgcn_s_memrealtime() - t0 > DEADLINE_TICKS ||
            __hip_atomic_load((unsigned*)(myws + WS_TMO), __ATOMIC_RELAXED, __HIP_MEMORY_SCOPE_AGENT) == seq) {
          if (lane == 0)
            __hip_atomic_store((unsigned*)(myws + WS_TMO), seq, __ATOMIC_RELAXED, __HIP_MEMORY_SCOPE_AGENT);
          lds_wr(CW(C_FAIL), 1);
          if (lane < NREM) lds_wr(CW(C_QUEUE + lane), (unsigned)(rank * 8));
          WAITL0();
          lds_wr(CW(C_RDY), NREM);
          cnt = NREM;
        }
      }
    }
    if (wg < 8) {
      const u64* gp = (const u64*)(myws + WS_AMAX) + wg * 512 + lane;
      bool ok = false;
      unsigned gl = 0, pass = 0, tmw = 0;
      for (;; ++pass) {
        tmw = __hip_atomic_load((unsigned*)(myws + WS_TMO), __ATOMIC_RELAXED, __HIP_MEMORY_SCOPE_AGENT);
        if (!ok) {
          ok = true; gl = 0;
#pragma unroll
          for (int k = 0; k < 8; ++k) {
            u64 x = __hip_atomic_load(gp + k * 64, __ATOMIC_RELAXED, __HIP_MEMORY_SCOPE_SYSTEM);
            ok &= (unsigned)(x >> 32) >= seq;
            gl = max(gl, (unsigned)x);
          }
        }
        if (__all(ok)) break;
        if ((pass & 31u) == 31u &&
            (__builtin_amdgcn_s_memrealtime() - t0 > DEADLINE_TICKS ||
             __hip_atomic_load((unsigned*)(myws + WS_TMO), __ATOMIC_RELAXED, __HIP_MEMORY_SCOPE_AGENT) == seq)) {
          if (lane == 0)
            __hip_atomic_store((unsigned*)(myws + WS_TMO), seq, __ATOMIC_RELAXED, __HIP_MEMORY_SCOPE_AGENT);
          gl = 0xFFFFFFFFu;
          break;
        }
      }
      if (tmw == seq) gl = 0xFFFFFFFFu;
      gl = wave_max_u32(gl);
      if (lane < 8)
        __hip_atomic_store((u64*)(myws + WS_FIN + lane * 128) + wg, ((u64)seq << 32) | gl, __ATOMIC_RELAXED, __HIP_MEMORY_SCOPE_AGENT);
    }
    return;
  }

  if (wave >= 4) {
    const int lw = wave - 4;
    const int r8 = lane >> 3;
    const int chA = (lane & 7) ^ r8;
    const unsigned offA_own = (unsigned)(tm * BM + lw * 32 + r8) * 1024u + chA * 16;
    const unsigned offA_stg = (unsigned)(tm * BM + lw * 32 + r8) * 128u + chA * 16;
    unsigned offB[4];
#pragma unroll
    for (int j = 0; j < 4; ++j) {
      const int k = (lw * 4 + j) * 4 + (lane >> 4);
      const int f = (k & 3) | (((k >> 3) & 1) << 2);
      const int sl = lane & 15;
      const int ch = (((sl >> 1) ^ f) << 1) | (sl & 1);
      offB[j] = (unsigned)k * (NDIM * 2) + tn * (BN * 2) + ch * 16;
    }
    const char* xown = (const char*)P.x + (size_t)rank * MLOC * 1024;
    const char* wsA = myws + WS_A;
    const char* wbase = (const char*)P.w;
    int li = 0, pub = 0, slot = 0, avail = NOWN, mind = 0, unext = 0;
    while (li < NSTEP) {
      while (li >= avail || li >= mind + NS) {
        if (li - pub == 2) { WAITV(8); ++pub; lds_wr(CW(C_FULL + lw), (unsigned)pub); }
        else if (li - pub == 1) { WAITV(0); ++pub; lds_wr(CW(C_FULL + lw), (unsigned)pub); }
        mind = lds_rd_min4(CW(C_DONE));
        avail = NOWN + __builtin_amdgcn_readfirstlane((int)lds_rd(CW(C_RDY)));
        unext = __builtin_amdgcn_readfirstlane((int)lds_rd(CW(C_QUEUE + (li >= NOWN ? li - NOWN : 0))));
      }
      int p, s;
      const char* abase;
      unsigned aoffs, astep;
      if (li < NOWN) {
        p = rank; s = 2 * li + (hk ^ (rank & 1));
        abase = xown + s * 128; aoffs = offA_own; astep = 8 * 1024;
      } else {
        p = unext >> 3; s = unext & 7;
        abase = wsA + (size_t)unext * 65536; aoffs = offA_stg; astep = 1024;
      }
      char* sb = smem + slot * STAGE;
#pragma unroll
      for (int j = 0; j < 4; ++j)
        glds16(abase + aoffs + j * astep, sb + (lw * 4 + j) * 1024 + lane * 16);
      const char* bb = wbase + (size_t)(p * KSH + s * 64) * (NDIM * 2);
#pragma unroll
      for (int j = 0; j < 4; ++j)
        glds16(bb + offB[j], sb + A_BYTES + (lw * 4 + j) * 1024 + lane * 16);
      ++li;
      slot = (slot == NS - 1) ? 0 : slot + 1;
      if (li == 1) bar();
      u32x4 vd; unsigned vr, vq;
      asm volatile("ds_read_b128 %0, %3\n\tds_read_b32 %1, %4\n\tds_read_b32 %2, %5"
                   : "=&v"(vd), "=&v"(vr), "=&v"(vq)
                   : "v"(CW(C_DONE)), "v"(CW(C_RDY)), "v"(CW(C_QUEUE + (li >= NOWN ? li - NOWN : 0))) : "memory");
      if (li - pub == 3) { WAITV(16); ++pub; lds_wr(CW(C_FULL + lw), (unsigned)pub); }
      asm volatile("s_waitcnt lgkmcnt(0)" : "+v"(vd), "+v"(vr), "+v"(vq) :: "memory");
      mind = __builtin_amdgcn_readfirstlane((int)min(min(vd[0], vd[1]), min(vd[2], vd[3])));
      avail = NOWN + __builtin_amdgcn_readfirstlane((int)vr);
      unext = __builtin_amdgcn_readfirstlane((int)vq);
    }
    WAITV(8); lds_wr(CW(C_FULL + lw), (unsigned)(NSTEP - 1));
    WAITV(0); lds_wr(CW(C_FULL + lw), (unsigned)NSTEP);
    return;
  }

  const int wm = wave >> 1, wn = wave & 1;
  const int fr = lane & 15, fq = lane >> 4;
  unsigned aoff[2];
#pragma unroll
  for (int kk = 0; kk < 2; ++kk)
    aoff[kk] = lds0 + (unsigned)(wm * 64 + fr) * 128u + (((kk * 4 + fq) ^ (lane & 7)) * 16);
  unsigned boff[4];
  {
    const int q = (lane & 15) >> 2, p = lane & 3, g = fq;
    const int f = q | ((g & 1) << 2);
#pragma unroll
    for (int nt = 0; nt < 4; ++nt)
      boff[nt] = lds0 + A_BYTES + (unsigned)(8 * g + q) * 256u + (((wn * 4 + nt) ^ f) * 32) + p * 8;
  }

  u64* gdst = (u64*)(P.ws[lane & (NR - 1)] + WS_AMAX) + (rank * NWG + wg) * 2 + wn;
  f32x4 acc[4][4];
#pragma unroll
  for (int i = 0; i < 4; ++i)
#pragma unroll
    for (int j = 0; j < 4; ++j) acc[i][j] = f32x4{0.f, 0.f, 0.f, 0.f};

  bar();
  if (wave == 0) STAMP(0);
  {
    bf16x8 a[4][2];
    s16x4 bl[4][2], bh[4][2];
#define WAIT_K(kk, n) asm volatile("s_waitcnt lgkmcnt(" #n ")" \
                   : "+v"(bl[0][kk]), "+v"(bh[0][kk]), "+v"(bl[1][kk]), "+v"(bh[1][kk]), \
                     "+v"(bl[2][kk]), "+v"(bh[2][kk]), "+v"(bl[3][kk]), "+v"(bh[3][kk]), \
                     "+v"(a[0][kk]), "+v"(a[1][kk]), "+v"(a[2][kk]), "+v"(a[3][kk]) :: "memory")
#define BFRAG(nt, kk) bf16x8{bl[nt][kk][0], bl[nt][kk][1], bl[nt][kk][2], bl[nt][kk][3], \
                             bh[nt][kk][0], bh[nt][kk][1], bh[nt][kk][2], bh[nt][kk][3]}
#define MMA(nt, kk) do { const bf16x8 bb_ = BFRAG(nt, kk); _Pragma("unroll") for (int mt = 0; mt < 4; ++mt) \
        acc[mt][nt] = __builtin_amdgcn_mfma_f32_16x16x32_bf16(bb_, a[mt][kk], acc[mt][nt], 0, 0, 0); } while (0)
    typedef short v4i16_t __attribute__((ext_vector_type(4)));
#define LDT(addr) __builtin_bit_cast(s16x4, __builtin_amdgcn_ds_read_tr16_b64_v4i16((LAS v4i16_t*)(size_t)(addr)))
#define LDA(addr) (*(const LAS bf16x8*)(size_t)(addr))
#define RDB(kk, sb_) do { _Pragma("unroll") for (int nt = 0; nt < 4; ++nt) { \
        bl[nt][kk] = LDT((sb_) + boff[nt] + (kk) * 8192); bh[nt][kk] = LDT((sb_) + boff[nt] + (kk) * 8192 + 1024); } \
        _Pragma("unroll") for (int mt = 0; mt < 4; ++mt) a[mt][kk] = LDA((sb_) + aoff[kk] + mt * 2048); } while (0)
#define ILV_PATTERN() do { _Pragma("unroll") for (int q = 0; q < 12; ++q) { \
        __builtin_amdgcn_sched_group_barrier(0x8, 1, 0); __builtin_amdgcn_sched_group_barrier(0x100, 1, 0); } \
        __builtin_amdgcn_sched_group_barrier(0x8, 4, 0); } while (0)
    int minf = 0;
    unsigned sb = 0;
    while (minf <= 0) minf = lds_rd_min4(CW(C_FULL));
    RDB(0, sb);
    __builtin_amdgcn_sched_barrier(0);
    for (int ci = 0; ci < NSTEP; ++ci) {
      RDB(1, sb);
      MMA(0, 0); MMA(1, 0); MMA(2, 0); MMA(3, 0);
      ILV_PATTERN();
      __builtin_amdgcn_sched_barrier(0);
      WAIT_K(1, 0);
      lds_wr(CW(C_DONE + wave), (unsigned)(ci + 1));
      sb = (sb == (NS - 1) * STAGE) ? 0u : sb + STAGE;
      if (ci + 1 < NSTEP) {
        while (minf <= ci + 1) minf = lds_rd_min4(CW(C_FULL));
      }
      __builtin_amdgcn_sched_barrier(0);
      RDB(0, sb);
      MMA(0, 1); MMA(1, 1); MMA(2, 1); MMA(3, 1);
      ILV_PATTERN();
      __builtin_amdgcn_sched_barrier(0);
    }
  }

  if (wave == 0) STAMP(2);
  const unsigned seq = (unsigned)__builtin_amdgcn_readfirstlane(lds_rd(CW(C_SEQ)));
  bool failed = lds_rd(CW(C_FAIL)) != 0;
  if (wm != hk) {
    if (wn == 0) STAMP(6);
    char* pb = myws + WS_PART + ((size_t)(wg * 2 + wn) * 16) * 1024 + lane * 16;
#pragma unroll
    for (int i = 0; i < 4; ++i)
#pragma unroll
      for (int j = 0; j < 4; ++j) {
        char* a_ = pb + (i * 4 + j) * 1024;
        asm volatile("global_store_dwordx4 %0, %1, off sc1\n\ts_nop 1" :: "v"(a_), "v"(acc[i][j]) : "memory");
      }
    WAITV(0);
    if (lane == 0)
      __hip_atomic_store((unsigned*)(myws + WS_XFLAG + (size_t)(wg * 2 + wn) * 128), seq, __ATOMIC_RELAXED, __HIP_MEMORY_SCOPE_AGENT);
    if (wn == 0) STAMP(7);
    return;
  }
  {
    const int pwg = wg ^ 8;
    const unsigned* xf = (const unsigned*)(myws + WS_XFLAG + (size_t)(pwg * 2 + wn) * 128);
    const unsigned long long t0 = __builtin_amdgcn_s_memrealtime();
    unsigned spins = 0;
    while (!failed && __hip_atomic_load(xf, __ATOMIC_RELAXED, __HIP_MEMORY_SCOPE_AGENT) < seq) {
      if ((++spins & 63u) != 0u) continue;
      if (__builtin_amdgcn_s_memrealtime() - t0 > DEADLINE_TICKS ||
          __hip_atomic_load((unsigned*)(myws + WS_TMO), __ATOMIC_RELAXED, __HIP_MEMORY_SCOPE_AGENT) == seq) {
        __hip_atomic_store((unsigned*)(myws + WS_TMO), seq, __ATOMIC_RELAXED, __HIP_MEMORY_SCOPE_AGENT);
        failed = true;
      }
    }
    if (wn == 0) STAMP(15);
    const char* pb = myws + WS_PART + ((size_t)(pwg * 2 + wn) * 16) * 1024 + lane * 16;
    f32x4 t[16];
#pragma unroll
    for (int i = 0; i < 16; ++i) {
      const char* a_ = pb + i * 1024;
      asm volatile("global_load_dwordx4 %0, %1, off sc1" : "=&v"(t[i]) : "v"(a_) : "memory");
    }
    asm volatile("s_waitcnt vmcnt(0)"
                 : "+v"(t[0]), "+v"(t[1]), "+v"(t[2]), "+v"(t[3]), "+v"(t[4]), "+v"(t[5]), "+v"(t[6]), "+v"(t[7]),
                   "+v"(t[8]), "+v"(t[9]), "+v"(t[10]), "+v"(t[11]), "+v"(t[12]), "+v"(t[13]), "+v"(t[14]), "+v"(t[15]) :: "memory");
#pragma unroll
    for (int i = 0; i < 4; ++i)
#pragma unroll
      for (int j = 0; j < 4; ++j) acc[i][j] += t[i * 4 + j];
  }
  if (wn == 0) STAMP(1);

  float m = 0.f;
#pragma unroll
  for (int i = 0; i < 4; ++i)
#pragma unroll
    for (int j = 0; j < 4; ++j)
#pragma unroll
      for (int r = 0; r < 4; ++r) m = fmaxf(m, fabsf(acc[i][j][r]));
  {
    const unsigned mw = wave_max_u32(__float_as_uint(m));
    if (lane < NR)
      __hip_atomic_store(gdst, ((u64)seq << 32) | mw, __ATOMIC_RELAXED, __HIP_MEMORY_SCOPE_SYSTEM);
  }
  if (wn == 0) STAMP(3);
  unsigned gm = 0;
  {
    const u64* fp = (const u64*)(myws + WS_FIN + (wg & 7) * 128) + (lane & 7);
    const unsigned long long t0 = __builtin_amdgcn_s_memrealtime();
    unsigned spins = 0;
    while (!failed) {
      const u64 x = __hip_atomic_load(fp, __ATOMIC_RELAXED, __HIP_MEMORY_SCOPE_AGENT);
      if (__all((unsigned)(x >> 32) >= seq)) { gm = wave_max_u32((unsigned)x); failed = gm == 0xFFFFFFFFu; break; }
      __builtin_amdgcn_s_sleep(2);
      if ((++spins & 63u) != 0u) continue;
      if (__builtin_amdgcn_s_memrealtime() - t0 > DEADLINE_TICKS ||
          __hip_atomic_load((unsigned*)(myws + WS_TMO), __ATOMIC_RELAXED, __HIP_MEMORY_SCOPE_AGENT) == seq) {
        __hip_atomic_store((unsigned*)(myws + WS_TMO), seq, __ATOMIC_RELAXED, __HIP_MEMORY_SCOPE_AGENT);
        failed = true;
      }
    }
  }
  const float amax = __uint_as_float(gm);
  if (wn == 0) STAMP(4);
  float scale = amax / 127.0f;
  if (!(scale > 0.f)) scale = 1.0f;
  const float rscale = 1.0f / scale;

  signed char* tile = (signed char*)smem + wn * 4096;
#pragma unroll
  for (int mt = 0; mt < 4; ++mt)
#pragma unroll
    for (int nt = 0; nt < 4; ++nt) {
      unsigned pk = 0;
#pragma unroll
      for (int r = 0; r < 4; ++r) {
        const float y = acc[mt][nt][r];
        float q0 = y * rscale;
        q0 = fmaf(fmaf(-q0, scale, y), rscale, q0);
        float qv = rintf(q0);
        qv = fminf(fmaxf(qv, -127.f), 127.f);
        int qi = failed ? -128 : (int)qv;
        pk |= ((unsigned)qi & 0xFFu) << (8 * r);
      }
      *(unsigned*)(tile + (mt * 16 + fr) * 64 + nt * 16 + fq * 4) = pk;
    }
  WAITL0();
  __builtin_amdgcn_wave_barrier();
#pragma unroll
  for (int j = 0; j < 4; ++j) {
    const int c = lane + j * 64;
    const int row = c >> 2, cc = c & 3;
    u32x4 v = *(const u32x4*)((const char*)tile + row * 64 + cc * 16);
    *(u32x4*)(P.out + (size_t)(tm * BM + hk * 64 + row) * NDIM + tn * BN + wn * 64 + cc * 16) = v;
  }
  if (wn == 0) STAMP(5);
}

}

extern "C" void kernel_launch(int rank, int nranks, ncclComm_t comm,
                              hipStream_t stream,
                              void* const* d_in, const int* in_sizes, int n_in,
                              void* d_out, int out_size,
                              const int* in_dtypes, int out_dtype,
                              void* const* d_out_all,
                              void* const* d_ws_all, size_t ws_bytes) {
  (void)comm; (void)in_sizes; (void)n_in; (void)out_size; (void)in_dtypes; (void)out_dtype; (void)d_out_all;
  if (nranks != NR || ws_bytes < WS_NEED || (unsigned)rank >= (unsigned)NR) {
    fprintf(stderr, "unexpected configuration\n");
    return;
  }
  Params P;
  P.x = (const unsigned short*)d_in[0];
  P.w = (const unsigned short*)d_in[1];
  P.out = (signed char*)d_out;
  for (int p = 0; p < NR; ++p) P.ws[p] = (char*)d_ws_all[p];
  P.rank = rank;
  hipLaunchKernelGGL(a2a_gemm_q8, dim3(NWG), dim3(NTHREADS), 0, stream, P);
  HIP_CHECK(hipGetLastError());
}
```
